```python
import numpy as np

kernel_name = "gpu_a2a_gemm_m4096_k4096_n4096_bf16_q8"


ABSMAX_RTOL = 0.02

COMPUTE_DTYPE = "bfloat16"
EPILOGUE = "quant_int8"
IN_QUANT = None
OUT_QUANT = "int8"
IN_DTYPES = {"x": "bfloat16", "w_mat": "bfloat16"}
WORLD_SIZE = 8

def _to_bf16(x):
    u = np.ascontiguousarray(x, dtype=np.float32).view(np.uint32)
    return ((u + 0x7FFF + ((u >> 16) & 1)) & 0xFFFF0000).view(np.float32)

def setup_inputs():

    x = _to_bf16(np.random.randn(4096, 4096).astype(np.float32))
    w_mat = _to_bf16((np.random.randn(4096, 4096) / np.sqrt(4096)).astype(np.float32))
    return {"x": x, "w_mat": w_mat}

def _quant_dequant(y, amax):
    scale = amax / 127.0
    q = np.clip(np.round(y / scale), -127.0, 127.0)
    return (q * scale).astype(np.float32)
def _epilogue(y):
    return _quant_dequant(y, np.abs(y).max())

def reference(x, w_mat):
    y = (x @ w_mat).astype(np.float32)
    return _epilogue(y)
```

```cpp
#include <hip/hip_runtime.h>
#include <rccl/rccl.h>
#include <cstdio>
#include <cstdint>

#define HIP_CHECK(expr) do { hipError_t _e = (expr); if (_e != hipSuccess) { \
    fprintf(stderr, "HIP error %s at %s:%d\n", hipGetErrorString(_e), __FILE__, __LINE__); \
    return; } } while (0)

namespace {

typedef short bf16x8 __attribute__((ext_vector_type(8)));
typedef short s16x4 __attribute__((ext_vector_type(4)));
typedef float f32x4 __attribute__((ext_vector_type(4)));
typedef unsigned u32x4 __attribute__((ext_vector_type(4)));

constexpr int NR = 8;
constexpr int MLOC = 512;
constexpr int KB = 512;
constexpr int NN = 4096;
constexpr int CH = 64;
constexpr int NCH = KB / CH;
constexpr int BM = 128, BN = 128;
constexpr int NST = NR * NCH / 2;
constexpr int NOWN = NCH / 2;
constexpr int NSLOT = 4;
constexpr int A_BYTES = BM * CH * 2;
constexpr int B_BYTES = CH * BN * 2;
constexpr int SLOT_BYTES = A_BYTES + B_BYTES;
constexpr int N_WG = 256;
constexpr int NRG = MLOC / 16;
constexpr int NT = 640;
#define PUSH_WIN 1

constexpr size_t WS_A = 0;
constexpr size_t WS_X = (size_t)NR * KB * MLOC * 2;
constexpr size_t WS_FLAG = WS_X + (size_t)N_WG * 32768;
constexpr size_t WS_XFLAG = WS_FLAG + NR * NCH * 4 * 128;
constexpr size_t WS_GRAN = WS_XFLAG + N_WG * 64;
constexpr size_t WS_LOCAL = WS_GRAN + 8 * 512;
constexpr size_t WS_EPOCH = WS_LOCAL + 2048;
constexpr size_t WS_TMO = WS_EPOCH + 128;
constexpr size_t WS_END = WS_TMO + 128;

constexpr unsigned long long DEADLINE_TICKS = 50000000ull;

struct Params {
    const char* x;
    const char* w;
    signed char* out;
    char* ws[NR];
    int rank;
};

__device__ __forceinline__ unsigned ld_agent32(const void* p) {
    return __hip_atomic_load((const unsigned*)p, __ATOMIC_RELAXED, __HIP_MEMORY_SCOPE_AGENT);
}
__device__ __forceinline__ unsigned long long ld_sys64(const void* p) {
    return __hip_atomic_load((const unsigned long long*)p, __ATOMIC_RELAXED, __HIP_MEMORY_SCOPE_SYSTEM);
}
__device__ __forceinline__ unsigned long long ld_agent64(const void* p) {
    return __hip_atomic_load((const unsigned long long*)p, __ATOMIC_RELAXED, __HIP_MEMORY_SCOPE_AGENT);
}

__device__ __forceinline__ bool give_up(unsigned* tmo, unsigned call, unsigned long long t0) {
    if (ld_agent32(tmo) == call || __builtin_amdgcn_s_memrealtime() - t0 > DEADLINE_TICKS) {
        __hip_atomic_store(tmo, call, __ATOMIC_RELAXED, __HIP_MEMORY_SCOPE_AGENT);
        return true;
    }
    return false;
}

__device__ __forceinline__ void store16_sys(void* p, u32x4 v) {
    asm volatile("global_store_dwordx4 %0, %1, off sc0 sc1\n\ts_nop 1" :: "v"(p), "v"(v) : "memory");
}
__device__ __forceinline__ void store16_agent(void* p, f32x4 v) {
    asm volatile("global_store_dwordx4 %0, %1, off sc1\n\ts_nop 1" :: "v"(p), "v"(v) : "memory");
}
__device__ __forceinline__ void dma16(unsigned voff, const char* sbase, unsigned lds) {
    asm volatile("s_mov_b32 m0, %2\n\ts_nop 0\n\tglobal_load_lds_dwordx4 %0, %1" :: "v"(voff), "s"(sbase), "s"(lds) : "memory");
}
__device__ __forceinline__ unsigned vm_outstanding() {
    unsigned s;
    asm volatile("s_getreg_b32 %0, hwreg(HW_REG_IB_STS)" : "=s"(s) :: "memory");
    return (s & 15u) | ((s >> 18) & 48u);
}

#define STAMP(i) do { } while (0)

__device__ __forceinline__ int swzB(int row) { return ((row & 3) << 2) | ((row >> 2) & 3); }

struct Frag { bf16x8 a[4]; s16x4 l[4], h[4]; };

__global__ void __launch_bounds__(NT, 3) a2a_gemm_q8(Params p) {
    __shared__ __attribute__((aligned(16))) char smem[NSLOT * SLOT_BYTES + 512];
    const int tid = threadIdx.x, lane = tid & 63;
    const int wave = __builtin_amdgcn_readfirstlane(tid >> 6);
    const int rank = p.rank;
    char* myws = p.ws[rank];
    unsigned* tmo = (unsigned*)(myws + WS_TMO);
    const unsigned long long t0 = __builtin_amdgcn_s_memrealtime();

    const int g = blockIdx.x;
    const int grp = g & 7, idx = g >> 3;
    const int h = idx & 1, nt = grp * 4 + ((idx >> 1) & 3), mt = idx >> 3;
    const unsigned smem_base = (unsigned)(size_t)smem;

    if (wave == 8) {
        const int j = g / NRG, rg = g % NRG;
        const int dst = (rank + 1 + j) & (NR - 1);
        const bool active = g < 7 * NRG && (unsigned)dst < (unsigned)NR;
        char* dws = p.ws[active ? dst : rank];
        const char* src = p.x + (size_t)((active ? dst : rank) * MLOC + rg * 16 + (lane >> 3)) * (KB * 2) + (lane & 7) * 16;
        char* dbase = dws + WS_A + ((size_t)rank * NCH * MLOC + rg * 16) * (CH * 2) + lane * 16;
        unsigned* flag = (unsigned*)(dws + WS_FLAG) + ((size_t)rank * NCH * 4 + (rg >> 3)) * 32 + (rg & 7);
        STAMP(0);
        u32x4 v[NCH][2];
#pragma unroll
        for (int kc = 0; kc < NCH; ++kc)
#pragma unroll
            for (int q = 0; q < 2; ++q)
                asm volatile("global_load_dwordx4 %0, %1, off" : "=v"(v[kc][q]) : "v"(src + (size_t)q * 8 * (KB * 2) + kc * (CH * 2)) : "memory");
        unsigned epoch;
        asm volatile("global_load_dword %0, %1, off sc1" : "=v"(epoch) : "v"(myws + WS_EPOCH) : "memory");
        int niss = active ? 0 : NCH, npub = active ? 0 : NCH;
        unsigned nops = 0;
        unsigned long long endq = 0;
        asm volatile("s_waitcnt vmcnt(13)" : "+v"(v[0][0]), "+v"(v[0][1]), "+v"(v[1][0]), "+v"(v[1][1]) :: "memory");
        if (active) {
            store16_sys(dbase, v[0][0]); store16_sys(dbase + 1024, v[0][1]);
            store16_sys(dbase + (size_t)MLOC * (CH * 2), v[1][0]); store16_sys(dbase + (size_t)MLOC * (CH * 2) + 1024, v[1][1]);
            nops = 4; endq = 2ull | (4ull << 8); niss = 2;
        }
        asm volatile("s_waitcnt vmcnt(4)"
                     : "+v"(v[2][0]), "+v"(v[2][1]), "+v"(v[3][0]), "+v"(v[3][1]), "+v"(v[4][0]), "+v"(v[4][1]), "+v"(v[5][0]), "+v"(v[5][1]),
                       "+v"(v[6][0]), "+v"(v[6][1]), "+v"(v[7][0]), "+v"(v[7][1]), "+v"(epoch) :: "memory");
        const unsigned call = __builtin_amdgcn_readfirstlane(epoch) + 1u;
        auto progress = [&]() {
            while (npub < niss && vm_outstanding() <= nops - (unsigned)((endq >> (8 * npub)) & 0xffull)) {
                if (lane == 0) __hip_atomic_store(flag + npub * 4 * 32, call, __ATOMIC_RELAXED, __HIP_MEMORY_SCOPE_SYSTEM);
                asm volatile("" ::: "memory");
                ++npub; ++nops;
            }
#pragma unroll
            for (int kc = 2; kc < NCH; ++kc)
                if (niss == kc && kc < npub + PUSH_WIN) {
                    store16_sys(dbase + (size_t)kc * MLOC * (CH * 2), v[kc][0]);
                    store16_sys(dbase + (size_t)kc * MLOC * (CH * 2) + 1024, v[kc][1]);
                    nops += 2; endq |= (unsigned long long)nops << (8 * kc); ++niss;
                }
        };
        progress();
        __builtin_amdgcn_s_barrier();
        for (int i = 0; i < NST; ++i) {
            const int st = i + NSLOT - 1;
            const int need = st >= NOWN && st < NST ? 2 * ((st - NOWN) / 7) + 2 : 0;
            progress();
            while (npub < need) { __builtin_amdgcn_s_sleep(1); progress(); }
            __builtin_amdgcn_s_barrier();
        }
        while (npub < NCH) { progress(); __builtin_amdgcn_s_sleep(1); }
        return;
    }

    if (wave == 9) {
        const unsigned call = ld_agent32(myws + WS_EPOCH) + 1u;
        asm volatile("s_waitcnt vmcnt(0)" ::: "memory");
        const int l = lane < 56 ? lane : 0;
        const unsigned* fbase = (const unsigned*)(myws + WS_FLAG) +
                                ((((rank + 1 + (l >> 3)) & (NR - 1)) * NCH + h) * 4 + mt) * 32 + (l & 7);
        unsigned long long rdy0 = 0, rdy1 = 0;
        int nr = 0, nj = 0;
        int pr = 0;
        bool pending = false, dead = false;
        const unsigned pbuf = smem_base + NSLOT * SLOT_BYTES;
#define POLL_ISSUE() do {   \
            const int r1 = nr + 1 < NCH / 2 ? nr + 1 : nr; \
            asm volatile("s_mov_b32 m0, %2\n\ts_nop 0\n\tglobal_load_lds_dword %0, off sc0 sc1\n\t" \
                         "s_mov_b32 m0, %3\n\ts_nop 0\n\tglobal_load_lds_dword %1, off sc0 sc1" \
                         :: "v"(fbase + nr * 2 * 4 * 32), "v"(fbase + r1 * 2 * 4 * 32), "s"(pbuf), "s"(pbuf + 256) : "memory"); \
            pending = true; pr = nr; } while (0)
#define POLL_TAKE() do { if (pending && vm_outstanding() == 0) { \
                unsigned f0, f1; \
                asm volatile("ds_read_b32 %0, %2\n\tds_read_b32 %1, %2 offset:256\n\ts_waitcnt lgkmcnt(0)" \
                             : "=&v"(f0), "=&v"(f1) : "v"(pbuf + 4 * lane) : "memory"); \
                const unsigned long long m0 = __ballot(f0 >= call), m1 = __ballot(f1 >= call); \
                rdy0 |= pr == nr ? m0 : (pr + 1 == nr ? m1 : 0ull); \
                rdy1 |= pr == nr ? m1 : 0ull; \
                pending = false; } } while (0)
        POLL_ISSUE();
        __builtin_amdgcn_s_barrier();
        for (int i = 0; i < NST; ++i) {
            const int st = i + NSLOT - 1;
            if (st >= NOWN && st < NST) {
                unsigned spins = 0;
                for (;;) {
                    POLL_TAKE();
                    if (((rdy0 >> (8 * nj)) & 0xffull) == 0xffull || dead) break;
                    if (!pending) {
                        if ((++spins & 15u) == 0u && give_up(tmo, call, t0)) dead = true;
                        else POLL_ISSUE();
                    }
                    __builtin_amdgcn_s_sleep(1);
                }
                if (++nj == 7) { nj = 0; ++nr; rdy0 = rdy1; rdy1 = 0; }
                if (!pending && nr < NCH / 2 && ((rdy0 >> (8 * nj)) & 0xffull) != 0xffull) POLL_ISSUE();
            }
            __builtin_amdgcn_s_barrier();
        }
#undef POLL_ISSUE
#undef POLL_TAKE
        return;
    }

    if (wave >= 4) {
        const int lw = wave - 4;
        unsigned avo_own[4], avo_rem[4], bvo[4];
#pragma unroll
        for (int i = 0; i < 4; ++i) {
            const int row = lw * 32 + i * 8 + (lane >> 3);
            const int ch = ((lane & 7) ^ ((row >> 1) & 7)) << 4;
            avo_own[i] = row * (KB * 2) + ch;
            avo_rem[i] = row * (CH * 2) + ch;
            const int brow = lw * 16 + i * 4 + (lane >> 4);
            bvo[i] = brow * (NN * 2) + (((lane & 15) ^ swzB(brow)) << 4);
        }
        const char* a_own = p.x + (size_t)(rank * MLOC + mt * BM) * (KB * 2) + h * (CH * 2);
        const char* a_rem = myws + WS_A + (size_t)(h * MLOC + mt * BM) * (CH * 2);
        const char* b0 = p.w + (size_t)h * CH * (NN * 2) + nt * (BN * 2);
        int islot = 0, ir = 0, ij = 0;
        auto issue = [&](int s) {
            const unsigned buf = smem_base + islot * SLOT_BYTES + lw * 4096;
            if (s < NOWN) {
                const char* ab = a_own + s * (2 * CH * 2);
                const char* bb = b0 + (size_t)(rank * KB + s * 2 * CH) * (NN * 2);
#pragma unroll
                for (int i = 0; i < 4; ++i) { dma16(avo_own[i], ab, buf + i * 1024); dma16(bvo[i], bb, buf + A_BYTES + i * 1024); }
            } else {
                const int src = (rank + 1 + ij) & (NR - 1);
                const char* ab = a_rem + (size_t)(src * NCH + 2 * ir) * MLOC * (CH * 2);
                const char* bb = b0 + (size_t)(src * KB + ir * 2 * CH) * (NN * 2);
#pragma unroll
                for (int i = 0; i < 4; ++i) { dma16(avo_rem[i], ab, buf + i * 1024); dma16(bvo[i], bb, buf + A_BYTES + i * 1024); }
                if (++ij == 7) { ij = 0; ++ir; }
            }
            islot = islot == NSLOT - 1 ? 0 : islot + 1;
        };
#pragma unroll
        for (int s = 0; s < NSLOT - 1; ++s) issue(s);
        asm volatile("s_waitcnt vmcnt(%0)" :: "n"(8 * (NSLOT - 2)) : "memory");
        __builtin_amdgcn_s_barrier();
        for (int i = 0; i < NST; ++i) {
            if (i + NSLOT - 2 < NST) asm volatile("s_waitcnt vmcnt(%0)" :: "n"(8 * (NSLOT - 3)) : "memory");
            else asm volatile("s_waitcnt vmcnt(0)" ::: "memory");
            __builtin_amdgcn_s_barrier();
            asm volatile("" ::: "memory");
            if (i + NSLOT - 1 < NST) issue(i + NSLOT - 1);
        }
        return;
    }

    STAMP(0);
    const unsigned call = ld_agent32(myws + WS_EPOCH) + 1u;
    const int wm = wave >> 1, wn = wave & 1;
    f32x4 acc[4][4];
#pragma unroll
    for (int a = 0; a < 4; ++a)
#pragma unroll
        for (int b = 0; b < 4; ++b) acc[a][b] = (f32x4){0.f, 0.f, 0.f, 0.f};
    const int fr = lane & 15, fq = lane >> 4;
    const int aoff0 = (wm * 64 + fr) * 128 + (((0 + fq) ^ ((fr >> 1) & 7)) << 4);
    const int aoff1 = (wm * 64 + fr) * 128 + (((4 + fq) ^ ((fr >> 1) & 7)) << 4);
    int boff[2][4];
#pragma unroll
    for (int t = 0; t < 2; ++t) {
        const int row = 8 * fq + 4 * t + ((lane & 15) >> 2);
#pragma unroll
        for (int nb = 0; nb < 4; ++nb)
            boff[t][nb] = A_BYTES + row * 256 + 8 * (lane & 1) + (((wn * 8 + 2 * nb + ((lane & 3) >> 1)) ^ swzB(row)) << 4);
    }
    Frag X, Y;
#define SB __builtin_amdgcn_sched_barrier(0)
#define R128(D, ADDR, OFF) asm volatile("ds_read_b128 %0, %1 offset:" #OFF : "=v"(D) : "v"(ADDR) : "memory")
#define RTR(D, ADDR, OFF) asm volatile("ds_read_b64_tr_b16 %0, %1 offset:" #OFF : "=v"(D) : "v"(ADDR) : "memory")
#define CAT(lo, hi) __builtin_shufflevector(lo, hi, 0, 1, 2, 3, 4, 5, 6, 7)
#define WL(F) asm volatile("s_waitcnt lgkmcnt(0)" : "+v"(F.a[0]), "+v"(F.a[1]), "+v"(F.a[2]), "+v"(F.a[3]), \
        "+v"(F.l[0]), "+v"(F.h[0]), "+v"(F.l[1]), "+v"(F.h[1]), "+v"(F.l[2]), "+v"(F.h[2]), "+v"(F.l[3]), "+v"(F.h[3]) :: "memory")
#define M1(I, J, F, A) acc[I][J] = __builtin_amdgcn_mfma_f32_16x16x32_bf16(F, A, acc[I][J], 0, 0, 0)
#define HALF(C, D, SBASE, AO, BO, NX) do { \
        const unsigned aa = (SBASE) + (AO), b00 = (SBASE) + boff[0][0], b01 = (SBASE) + boff[0][1], b02 = (SBASE) + boff[0][2], b03 = (SBASE) + boff[0][3]; \
        const unsigned b10 = (SBASE) + boff[1][0], b11 = (SBASE) + boff[1][1], b12 = (SBASE) + boff[1][2], b13 = (SBASE) + boff[1][3]; \
        const bf16x8 f0 = CAT(C.l[0], C.h[0]), f1 = CAT(C.l[1], C.h[1]), f2 = CAT(C.l[2], C.h[2]), f3 = CAT(C.l[3], C.h[3]); \
        SB; M1(0, 0, f0, C.a[0]); SB; if (NX) { RTR(D.l[0], b00, BO); RTR(D.h[0], b10, BO); } \
        SB; M1(0, 1, f1, C.a[0]); SB; if (NX) { R128(D.a[0], aa, 0); } \
        SB; M1(0, 2, f2, C.a[0]); SB; if (NX) { RTR(D.l[1], b01, BO); RTR(D.h[1], b11, BO); } \
        SB; M1(0, 3, f3, C.a[0]); SB; if (NX) { R128(D.a[1], aa, 2048); } \
        SB; M1(1, 0, f0, C.a[1]); SB; if (NX) { RTR(D.l[2], b02, BO); RTR(D.h[2], b12, BO); } \
        SB; M1(1, 1, f1, C.a[1]); SB; if (NX) { R128(D.a[2], aa, 4096); } \
        SB; M1(1, 2, f2, C.a[1]); SB; if (NX) { RTR(D.l[3], b03, BO); RTR(D.h[3], b13, BO); } \
        SB; M1(1, 3, f3, C.a[1]); SB; if (NX) { R128(D.a[3], aa, 6144); } \
        SB; M1(2, 0, f0, C.a[2]); M1(2, 1, f1, C.a[2]); M1(2, 2, f2, C.a[2]); M1(2, 3, f3, C.a[2]); \
        M1(3, 0, f0, C.a[3]); M1(3, 1, f1, C.a[3]); M1(3, 2, f2, C.a[3]); M1(3, 3, f3, C.a[3]); SB; } while (0)
    __builtin_amdgcn_s_barrier();
    asm volatile("" ::: "memory");
    {
        const unsigned sb = smem_base;
#pragma unroll
        for (int nb = 0; nb < 4; ++nb) { RTR(X.l[nb], sb + boff[0][nb], 0); RTR(X.h[nb], sb + boff[1][nb], 0); }
        const unsigned aa = sb + aoff0;
        R128(X.a[0], aa, 0); R128(X.a[1], aa, 2048); R128(X.a[2], aa, 4096); R128(X.a[3], aa, 6144);
    }
    int slot = 0;
    for (int i = 0; i < NST; ++i) {
        WL(X);
        __builtin_amdgcn_s_barrier();
        asm volatile("" ::: "memory");
        const unsigned sb = smem_base + slot * SLOT_BYTES;
        const int nslot = slot == NSLOT - 1 ? 0 : slot + 1;
        const unsigned sn = smem_base + nslot * SLOT_BYTES;
        HALF(X, Y, sb, aoff1, 8192, true);
        WL(Y);
        HALF(Y, X, sn, aoff0, 0, i + 1 < NST);
        slot = nslot;
    }
#undef HALF
#undef M1
#undef WL
#undef CAT
#undef RTR
#undef R128
#undef SB
    if (wave == 0) STAMP(10);

    f32x4 keep[2][4];
    {
        char* xb = myws + WS_X + ((size_t)g * 4 + wave) * 8192 + lane * 16;
#pragma unroll
        for (int a = 0; a < 2; ++a)
#pragma unroll
            for (int b = 0; b < 4; ++b) {
                store16_agent(xb + (a * 4 + b) * 1024, h ? acc[a][b] : acc[2 + a][b]);
                keep[a][b] = h ? acc[2 + a][b] : acc[a][b];
            }
        asm volatile("s_waitcnt vmcnt(0)" ::: "memory");
    }
    __syncthreads();
    if (tid == 0)
        __hip_atomic_store((unsigned*)(myws + WS_XFLAG + g * 64), call, __ATOMIC_RELAXED, __HIP_MEMORY_SCOPE_AGENT);
    float* red = (float*)(smem + 16384);
    {
        const int gp = g ^ 8;
        while (ld_agent32(myws + WS_XFLAG + gp * 64) != call) {
            if (give_up(tmo, call, t0)) break;
            __builtin_amdgcn_s_sleep(1);
        }
        const char* xb = myws + WS_X + ((size_t)gp * 4 + wave) * 8192 + lane * 16;
        f32x4 r0, r1, r2, r3, r4, r5, r6, r7;
        asm volatile("global_load_dwordx4 %0, %8, off sc1\n\t"
                     "global_load_dwordx4 %1, %8, off offset:1024 sc1\n\t"
                     "global_load_dwordx4 %2, %8, off offset:2048 sc1\n\t"
                     "global_load_dwordx4 %3, %8, off offset:3072 sc1\n\t"
                     "global_load_dwordx4 %4, %9, off sc1\n\t"
                     "global_load_dwordx4 %5, %9, off offset:1024 sc1\n\t"
                     "global_load_dwordx4 %6, %9, off offset:2048 sc1\n\t"
                     "global_load_dwordx4 %7, %9, off offset:3072 sc1\n\t"
                     "s_waitcnt vmcnt(0)"
                     : "=&v"(r0), "=&v"(r1), "=&v"(r2), "=&v"(r3), "=&v"(r4), "=&v"(r5), "=&v"(r6), "=&v"(r7)
                     : "v"(xb), "v"(xb + 4096) : "memory");
        keep[0][0] += r0; keep[0][1] += r1; keep[0][2] += r2; keep[0][3] += r3;
        keep[1][0] += r4; keep[1][1] += r5; keep[1][2] += r6; keep[1][3] += r7;
        float m = 0.f;
#pragma unroll
        for (int a = 0; a < 2; ++a)
#pragma unroll
            for (int b = 0; b < 4; ++b)
#pragma unroll
                for (int r = 0; r < 4; ++r) m = fmaxf(m, fabsf(keep[a][b][r]));
#pragma unroll
        for (int o = 32; o > 0; o >>= 1) m = fmaxf(m, __shfl_xor(m, o));
        if (lane == 0) red[wave] = m;
    }
    __syncthreads();
    if (wave == 0) STAMP(11);

    if (wave == 0) {
        const float wgm = fmaxf(fmaxf(red[0], red[1]), fmaxf(red[2], red[3]));
        const unsigned long long tag = (unsigned long long)call << 32;
        unsigned long long* loc = (unsigned long long*)(myws + WS_LOCAL);
        if (lane == 0)
            __hip_atomic_store(loc + g, tag | __float_as_uint(wgm), __ATOMIC_RELAXED, __HIP_MEMORY_SCOPE_AGENT);
        if (idx == 0) {
            unsigned long long v;
            for (;;) {
                v = ld_agent64(loc + grp + 8 * (lane & 31));
                if (__all((unsigned)(v >> 32) == call)) break;
                if (give_up(tmo, call, t0)) break;
                __builtin_amdgcn_s_sleep(1);
            }
            unsigned gm = (unsigned)v;
#pragma unroll
            for (int o = 32; o > 0; o >>= 1) { unsigned x = __shfl_xor(gm, o); gm = x > gm ? x : gm; }
            __hip_atomic_store((unsigned long long*)(p.ws[lane >> 3] + WS_GRAN) + (lane & 7) * 64 + rank * 8 + grp, tag | gm,
                               __ATOMIC_RELAXED, __HIP_MEMORY_SCOPE_SYSTEM);
        }
        unsigned long long v;
        const unsigned long long* gran = (const unsigned long long*)(myws + WS_GRAN) + grp * 64;
        for (;;) {
            v = ld_sys64(gran + lane);
            if (__all((unsigned)(v >> 32) == call)) break;
            if (give_up(tmo, call, t0)) break;
            __builtin_amdgcn_s_sleep(1);
        }
        unsigned gm = (unsigned)v;
#pragma unroll
        for (int o = 32; o > 0; o >>= 1) { unsigned x = __shfl_xor(gm, o); gm = x > gm ? x : gm; }
        if (lane == 0) {
            red[4] = __uint_as_float(gm);
            ((unsigned*)red)[5] = (ld_agent32(tmo) == call) ? 1u : 0u;
            if (g == 0) __hip_atomic_store((unsigned*)(myws + WS_EPOCH), call, __ATOMIC_RELAXED, __HIP_MEMORY_SCOPE_AGENT);
        }
    }
    __syncthreads();
    if (wave == 0) STAMP(12);
    const float amax = red[4];
    const bool bad = ((unsigned*)red)[5] != 0u;
    const float inv = amax > 0.f ? 127.f / amax : 0.f;

    constexpr int OSTR = BN + 16;
    char* ot = smem;
#pragma unroll
    for (int mb = 0; mb < 2; ++mb)
#pragma unroll
        for (int nb = 0; nb < 4; ++nb) {
            unsigned pk = 0;
#pragma unroll
            for (int r = 0; r < 4; ++r) {
                float q = rintf(keep[mb][nb][r] * inv);
                q = fminf(fmaxf(q, -127.f), 127.f);
                int qi = bad ? -128 : (int)q;
                pk |= ((unsigned)qi & 0xffu) << (8 * r);
            }
            const int row = wm * 32 + mb * 16 + fr, col = wn * 64 + nb * 16 + fq * 4;
            *(unsigned*)(ot + row * OSTR + col) = pk;
        }
    __syncthreads();
#pragma unroll
    for (int i = 0; i < 2; ++i) {
        const int row = (tid >> 3) + 32 * i, c16 = tid & 7;
        const u32x4 v = *(const u32x4*)(ot + row * OSTR + c16 * 16);
        *(u32x4*)(p.out + (size_t)(mt * BM + i * 64 + h * 32 + (tid >> 3)) * NN + nt * BN + c16 * 16) = v;
    }
    if (wave == 0) STAMP(13);
}

}

extern "C" void kernel_launch(int rank, int nranks, ncclComm_t comm,
                              hipStream_t stream,
                              void* const* d_in, const int* in_sizes, int n_in,
                              void* d_out, int out_size,
                              const int* in_dtypes, int out_dtype,
                              void* const* d_out_all,
                              void* const* d_ws_all, size_t ws_bytes) {
    if (nranks != NR || n_in < 2 || ws_bytes < WS_END || (unsigned)rank >= (unsigned)NR) {
        fprintf(stderr, "unsupported configuration\n");
        return;
    }
    Params p;
    p.x = (const char*)d_in[0];
    p.w = (const char*)d_in[1];
    p.out = (signed char*)d_out;
    for (int i = 0; i < NR; ++i) p.ws[i] = (char*)d_ws_all[i];
    p.rank = rank;
    hipLaunchKernelGGL(a2a_gemm_q8, dim3(N_WG), dim3(NT), 0, stream, p);
    HIP_CHECK(hipGetLastError());
}
```

```cpp
#include <hip/hip_runtime.h>
#include <rccl/rccl.h>
#include <cstdio>
#include <cstdint>

#define HIP_CHECK(expr) do { hipError_t _e = (expr); if (_e != hipSuccess) { \
    fprintf(stderr, "HIP error %s at %s:%d\n", hipGetErrorString(_e), __FILE__, __LINE__); \
    return; } } while (0)

namespace {

typedef short bf16x8 __attribute__((ext_vector_type(8)));
typedef short s16x4 __attribute__((ext_vector_type(4)));
typedef float f32x4 __attribute__((ext_vector_type(4)));
typedef unsigned u32x4 __attribute__((ext_vector_type(4)));

constexpr int NR = 8;
constexpr int MLOC = 512;
constexpr int KB = 512;
constexpr int NN = 4096;
constexpr int CH = 64;
constexpr int NCH = KB / CH;
constexpr int BM = 128, BN = 128;
constexpr int NST = NR * NCH / 2;
constexpr int NOWN = NCH / 2;
constexpr int NSLOT = 4;
constexpr int A_BYTES = BM * CH * 2;
constexpr int B_BYTES = CH * BN * 2;
constexpr int SLOT_BYTES = A_BYTES + B_BYTES;
constexpr int N_WG = 256;
constexpr int NRG = MLOC / 16;
constexpr int NT = 640;
#define POLL_LOW 3
#define PUSH_WIN 1

constexpr size_t WS_A = 0;
constexpr size_t WS_X = (size_t)NR * KB * MLOC * 2;
constexpr size_t WS_FLAG = WS_X + (size_t)N_WG * 32768;
constexpr size_t WS_XFLAG = WS_FLAG + NR * NCH * 4 * 128;
constexpr size_t WS_GRAN = WS_XFLAG + N_WG * 64;
constexpr size_t WS_LOCAL = WS_GRAN + 8 * 512;
constexpr size_t WS_EPOCH = WS_LOCAL + 2048;
constexpr size_t WS_TMO = WS_EPOCH + 128;
constexpr size_t WS_END = WS_TMO + 128;

constexpr unsigned long long DEADLINE_TICKS = 50000000ull;

struct Params {
    const char* x;
    const char* w;
    signed char* out;
    char* ws[NR];
    int rank;
};

__device__ __forceinline__ unsigned ld_agent32(const void* p) {
    return __hip_atomic_load((const unsigned*)p, __ATOMIC_RELAXED, __HIP_MEMORY_SCOPE_AGENT);
}
__device__ __forceinline__ unsigned long long ld_sys64(const void* p) {
    return __hip_atomic_load((const unsigned long long*)p, __ATOMIC_RELAXED, __HIP_MEMORY_SCOPE_SYSTEM);
}
__device__ __forceinline__ unsigned long long ld_agent64(const void* p) {
    return __hip_atomic_load((const unsigned long long*)p, __ATOMIC_RELAXED, __HIP_MEMORY_SCOPE_AGENT);
}

__device__ __forceinline__ bool give_up(unsigned* tmo, unsigned call, unsigned long long t0) {
    if (ld_agent32(tmo) == call || __builtin_amdgcn_s_memrealtime() - t0 > DEADLINE_TICKS) {
        __hip_atomic_store(tmo, call, __ATOMIC_RELAXED, __HIP_MEMORY_SCOPE_AGENT);
        return true;
    }
    return false;
}

__device__ __forceinline__ void store16_sys(void* p, u32x4 v) {
    asm volatile("global_store_dwordx4 %0, %1, off sc0 sc1\n\ts_nop 1" :: "v"(p), "v"(v) : "memory");
}
__device__ __forceinline__ void store16_agent(void* p, f32x4 v) {
    asm volatile("global_store_dwordx4 %0, %1, off sc1\n\ts_nop 1" :: "v"(p), "v"(v) : "memory");
}
__device__ __forceinline__ void dma16(unsigned voff, const char* sbase, unsigned lds) {
    asm volatile("s_mov_b32 m0, %2\n\ts_nop 0\n\tglobal_load_lds_dwordx4 %0, %1" :: "v"(voff), "s"(sbase), "s"(lds) : "memory");
}
__device__ __forceinline__ unsigned vm_outstanding() {
    unsigned s;
    asm volatile("s_getreg_b32 %0, hwreg(HW_REG_IB_STS)" : "=s"(s) :: "memory");
    return (s & 15u) | ((s >> 18) & 48u);
}

#define STAMP(i) do { } while (0)

__device__ __forceinline__ int swzB(int row) { return ((row & 3) << 2) | ((row >> 2) & 3); }

struct Frag { bf16x8 a[4]; s16x4 l[4], h[4]; };

__global__ void __launch_bounds__(NT, 3) a2a_gemm_q8(Params p) {
    __shared__ __attribute__((aligned(16))) char smem[NSLOT * SLOT_BYTES + 64 + 128 + 1024];
    const int tid = threadIdx.x, lane = tid & 63;
    const int wave = __builtin_amdgcn_readfirstlane(tid >> 6);
    const int rank = p.rank;
    char* myws = p.ws[rank];
    unsigned* tmo = (unsigned*)(myws + WS_TMO);
    const unsigned long long t0 = __builtin_amdgcn_s_memrealtime();

    const int g = blockIdx.x;
    const int grp = g & 7, idx = g >> 3;
    const int h = idx & 1, nt = grp * 4 + ((idx >> 1) & 3), mt = idx >> 3;
    const unsigned smem_base = (unsigned)(size_t)smem;
    const unsigned relw = smem_base + NSLOT * SLOT_BYTES;
    int c = 0, iss = NSLOT - 1, rel = NOWN, it = 0;
#define MACHINE_STEP(ACTIVE, CAN) \
        const bool ACTIVE = c + 1 < iss || c + 1 >= NST; \
        const int CAN = max(0, min(min(rel, c + NSLOT), NST) - iss);
#define MACHINE_NEXT(ACTIVE, CAN) do { rel = __builtin_amdgcn_readfirstlane(relv); iss += CAN; c += ACTIVE ? 1 : 0; ++it; } while (0)

    if (wave == 8) {
        const int j = g / NRG, rg = g % NRG;
        const int dst = (rank + 1 + j) & (NR - 1);
        const bool active = g < 7 * NRG && (unsigned)dst < (unsigned)NR;
        char* dws = p.ws[active ? dst : rank];
        const char* src = p.x + (size_t)((active ? dst : rank) * MLOC + rg * 16 + (lane >> 3)) * (KB * 2) + (lane & 7) * 16;
        char* dbase = dws + WS_A + ((size_t)rank * NCH * MLOC + rg * 16) * (CH * 2) + lane * 16;
        unsigned* flag = (unsigned*)(dws + WS_FLAG) + ((size_t)rank * NCH * 4 + (rg >> 3)) * 32 + (rg & 7);
        STAMP(0);
        u32x4 v[NCH][2];
#pragma unroll
        for (int kc = 0; kc < NCH; ++kc)
#pragma unroll
            for (int q = 0; q < 2; ++q)
                asm volatile("global_load_dwordx4 %0, %1, off" : "=v"(v[kc][q]) : "v"(src + (size_t)q * 8 * (KB * 2) + kc * (CH * 2)) : "memory");
        unsigned epoch;
        asm volatile("global_load_dword %0, %1, off sc1" : "=v"(epoch) : "v"(myws + WS_EPOCH) : "memory");
        int niss = active ? 0 : NCH, npub = active ? 0 : NCH;
        unsigned nops = 0;
        unsigned long long endq = 0;
        asm volatile("s_waitcnt vmcnt(13)" : "+v"(v[0][0]), "+v"(v[0][1]), "+v"(v[1][0]), "+v"(v[1][1]) :: "memory");
        if (active) {
            store16_sys(dbase, v[0][0]); store16_sys(dbase + 1024, v[0][1]);
            store16_sys(dbase + (size_t)MLOC * (CH * 2), v[1][0]); store16_sys(dbase + (size_t)MLOC * (CH * 2) + 1024, v[1][1]);
            nops = 4; endq = 2ull | (4ull << 8); niss = 2;
        }
        asm volatile("s_waitcnt vmcnt(4)"
                     : "+v"(v[2][0]), "+v"(v[2][1]), "+v"(v[3][0]), "+v"(v[3][1]), "+v"(v[4][0]), "+v"(v[4][1]), "+v"(v[5][0]), "+v"(v[5][1]),
                       "+v"(v[6][0]), "+v"(v[6][1]), "+v"(v[7][0]), "+v"(v[7][1]), "+v"(epoch) :: "memory");
        const unsigned call = __builtin_amdgcn_readfirstlane(epoch) + 1u;
        auto progress = [&]() {
            while (npub < niss && vm_outstanding() <= nops - (unsigned)((endq >> (8 * npub)) & 0xffull)) {
                if (lane == 0) __hip_atomic_store(flag + npub * 4 * 32, call, __ATOMIC_RELAXED, __HIP_MEMORY_SCOPE_SYSTEM);
                asm volatile("" ::: "memory");
                ++npub; ++nops;
            }
#pragma unroll
            for (int kc = 2; kc < NCH; ++kc)
                if (niss == kc && kc < npub + PUSH_WIN) {
                    store16_sys(dbase + (size_t)kc * MLOC * (CH * 2), v[kc][0]);
                    store16_sys(dbase + (size_t)kc * MLOC * (CH * 2) + 1024, v[kc][1]);
                    nops += 2; endq |= (unsigned long long)nops << (8 * kc); ++niss;
                }
        };
        progress();
        __builtin_amdgcn_s_barrier();
        while (c < NST && npub < NCH) {
            progress();
            __builtin_amdgcn_s_barrier();
            asm volatile("" ::: "memory");
            int relv = NST;
            if (rel < NST) asm volatile("ds_read_b32 %0, %1\n\ts_waitcnt lgkmcnt(0)" : "=v"(relv) : "v"(relw + 4 * (it & 1)) : "memory");
            MACHINE_STEP(act, can);
            MACHINE_NEXT(act, can);
        }
        while (npub < NCH) { progress(); __builtin_amdgcn_s_sleep(1); }
        return;
    }

    if (wave == 9) {
        const unsigned call = ld_agent32(myws + WS_EPOCH) + 1u;
        asm volatile("s_waitcnt vmcnt(0)" ::: "memory");
        const int l = lane < 56 ? lane : 0;
        const unsigned* fbase = (const unsigned*)(myws + WS_FLAG) +
                                ((((rank + 1 + (l >> 3)) & (NR - 1)) * NCH + h) * 4 + mt) * 32 + (l & 7);
        unsigned todo = (1u << 28) - 1u;
        int prel = NOWN;
        bool pending = false;
        const unsigned ordw = relw + 64, pbuf = relw + 64 + 128;
#define POLL_ISSUE() do { \
            asm volatile("s_mov_b32 m0, %4\n\ts_nop 0\n\tglobal_load_lds_dword %0, off sc0 sc1\n\t" \
                         "s_mov_b32 m0, %5\n\ts_nop 0\n\tglobal_load_lds_dword %1, off sc0 sc1\n\t" \
                         "s_mov_b32 m0, %6\n\ts_nop 0\n\tglobal_load_lds_dword %2, off sc0 sc1\n\t" \
                         "s_mov_b32 m0, %7\n\ts_nop 0\n\tglobal_load_lds_dword %3, off sc0 sc1" \
                         :: "v"(fbase), "v"(fbase + 256), "v"(fbase + 512), "v"(fbase + 768), \
                            "s"(pbuf), "s"(pbuf + 256), "s"(pbuf + 512), "s"(pbuf + 768) : "memory"); \
            pending = true; } while (0)
        POLL_ISSUE();
        __builtin_amdgcn_s_barrier();
        while (c < NST && rel < NST) {
            if (pending && vm_outstanding() == 0) {
                unsigned f0, f1, f2, f3;
                asm volatile("ds_read_b32 %0, %4\n\tds_read_b32 %1, %4 offset:256\n\tds_read_b32 %2, %4 offset:512\n\t"
                             "ds_read_b32 %3, %4 offset:768\n\ts_waitcnt lgkmcnt(0)"
                             : "=&v"(f0), "=&v"(f1), "=&v"(f2), "=&v"(f3) : "v"(pbuf + 4 * lane) : "memory");
                const unsigned long long m[4] = {__ballot(f0 >= call), __ballot(f1 >= call), __ballot(f2 >= call), __ballot(f3 >= call)};
                pending = false;
#pragma unroll
                for (int r = 0; r < 4; ++r)
#pragma unroll
                    for (int jj = 0; jj < 7; ++jj)
                        if (((todo >> (7 * r + jj)) & 1u) && ((m[r] >> (8 * jj)) & 0xffull) == 0xffull) {
                            asm volatile("ds_write_b32 %0, %1" :: "v"(ordw + 4 * prel), "v"(8 * r + jj) : "memory");
                            todo &= ~(1u << (7 * r + jj)); ++prel;
                        }
            }
            if (prel < NST && !pending && prel - iss < POLL_LOW) {
                if ((it & 63) == 63 && give_up(tmo, call, t0)) {
                    for (; prel < NST; ++prel) asm volatile("ds_write_b32 %0, %1" :: "v"(ordw + 4 * prel), "v"(0) : "memory");
                } else POLL_ISSUE();
            }
            asm volatile("ds_write_b32 %0, %1\n\ts_waitcnt lgkmcnt(0)" :: "v"(relw + 4 * (it & 1)), "v"(prel) : "memory");
            __builtin_amdgcn_s_barrier();
            asm volatile("" ::: "memory");
            const int relv = prel;
            MACHINE_STEP(act, can);
            MACHINE_NEXT(act, can);
        }
#undef POLL_ISSUE
        asm volatile("s_waitcnt vmcnt(0)" ::: "memory");
        return;
    }

    if (wave >= 4) {
        const int lw = wave - 4;
        unsigned avo_own[4], avo_rem[4], bvo[4];
#pragma unroll
        for (int i = 0; i < 4; ++i) {
            const int row = lw * 32 + i * 8 + (lane >> 3);
            const int ch = ((lane & 7) ^ ((row >> 1) & 7)) << 4;
            avo_own[i] = row * (KB * 2) + ch;
            avo_rem[i] = row * (CH * 2) + ch;
            const int brow = lw * 16 + i * 4 + (lane >> 4);
            bvo[i] = brow * (NN * 2) + (((lane & 15) ^ swzB(brow)) << 4);
        }
        const char* a_own = p.x + (size_t)(rank * MLOC + mt * BM) * (KB * 2) + h * (CH * 2);
        const char* a_rem = myws + WS_A + (size_t)(h * MLOC + mt * BM) * (CH * 2);
        const char* b0 = p.w + (size_t)h * CH * (NN * 2) + nt * (BN * 2);
        int islot = 0;
        auto issue = [&](int s) {
            const unsigned buf = smem_base + islot * SLOT_BYTES + lw * 4096;
            if (s < NOWN) {
                const char* ab = a_own + s * (2 * CH * 2);
                const char* bb = b0 + (size_t)(rank * KB + s * 2 * CH) * (NN * 2);
#pragma unroll
                for (int i = 0; i < 4; ++i) { dma16(avo_own[i], ab, buf + i * 1024); dma16(bvo[i], bb, buf + A_BYTES + i * 1024); }
            } else {
                int o;
                asm volatile("ds_read_b32 %0, %1\n\ts_waitcnt lgkmcnt(0)" : "=v"(o) : "v"(relw + 64 + 4 * s) : "memory");
                o = __builtin_amdgcn_readfirstlane(o);
                const int ir = (o >> 3) & 3, src = (rank + 1 + (o & 7)) & (NR - 1);
                const char* ab = a_rem + (size_t)(src * NCH + 2 * ir) * MLOC * (CH * 2);
                const char* bb = b0 + (size_t)(src * KB + ir * 2 * CH) * (NN * 2);
#pragma unroll
                for (int i = 0; i < 4; ++i) { dma16(avo_rem[i], ab, buf + i * 1024); dma16(bvo[i], bb, buf + A_BYTES + i * 1024); }
            }
            islot = islot == NSLOT - 1 ? 0 : islot + 1;
        };
#pragma unroll
        for (int s = 0; s < NSLOT - 1; ++s) issue(s);
        asm volatile("s_waitcnt vmcnt(%0)" :: "n"(8 * (NSLOT - 2)) : "memory");
        __builtin_amdgcn_s_barrier();
        while (c < NST) {
            if (iss >= c + 4) asm volatile("s_waitcnt vmcnt(16)" ::: "memory");
            else if (iss == c + 3) asm volatile("s_waitcnt vmcnt(8)" ::: "memory");
            else asm volatile("s_waitcnt vmcnt(0)" ::: "memory");
            __builtin_amdgcn_s_barrier();
            asm volatile("" ::: "memory");
            MACHINE_STEP(act, can);
            for (int k = 0; k < can; ++k) issue(iss + k);
            int relv = NST;
            if (rel < NST) asm volatile("ds_read_b32 %0, %1\n\ts_waitcnt lgkmcnt(0)" : "=v"(relv) : "v"(relw + 4 * (it & 1)) : "memory");
            MACHINE_NEXT(act, can);
        }
        return;
    }

    STAMP(0);
    const unsigned call = ld_agent32(myws + WS_EPOCH) + 1u;
    const int wm = wave >> 1, wn = wave & 1;
    f32x4 acc[4][4];
#pragma unroll
    for (int a = 0; a < 4; ++a)
#pragma unroll
        for (int b = 0; b < 4; ++b) acc[a][b] = (f32x4){0.f, 0.f, 0.f, 0.f};
    const int fr = lane & 15, fq = lane >> 4;
    const int aoff0 = (wm * 64 + fr) * 128 + (((0 + fq) ^ ((fr >> 1) & 7)) << 4);
    const int aoff1 = (wm * 64 + fr) * 128 + (((4 + fq) ^ ((fr >> 1) & 7)) << 4);
    int boff[2][4];
#pragma unroll
    for (int t = 0; t < 2; ++t) {
        const int row = 8 * fq + 4 * t + ((lane & 15) >> 2);
#pragma unroll
        for (int nb = 0; nb < 4; ++nb)
            boff[t][nb] = A_BYTES + row * 256 + 8 * (lane & 1) + (((wn * 8 + 2 * nb + ((lane & 3) >> 1)) ^ swzB(row)) << 4);
    }
    Frag X, Y;
#define SB __builtin_amdgcn_sched_barrier(0)
#define R128(D, ADDR, OFF) asm volatile("ds_read_b128 %0, %1 offset:" #OFF : "=v"(D) : "v"(ADDR) : "memory")
#define RTR(D, ADDR, OFF) asm volatile("ds_read_b64_tr_b16 %0, %1 offset:" #OFF : "=v"(D) : "v"(ADDR) : "memory")
#define CAT(lo, hi) __builtin_shufflevector(lo, hi, 0, 1, 2, 3, 4, 5, 6, 7)
#define WL(F) asm volatile("s_waitcnt lgkmcnt(0)" : "+v"(F.a[0]), "+v"(F.a[1]), "+v"(F.a[2]), "+v"(F.a[3]), \
        "+v"(F.l[0]), "+v"(F.h[0]), "+v"(F.l[1]), "+v"(F.h[1]), "+v"(F.l[2]), "+v"(F.h[2]), "+v"(F.l[3]), "+v"(F.h[3]) :: "memory")
#define M1(I, J, F, A) acc[I][J] = __builtin_amdgcn_mfma_f32_16x16x32_bf16(F, A, acc[I][J], 0, 0, 0)
#define HALF(C, D, SBASE, AO, BO, NX) do { \
        const unsigned aa = (SBASE) + (AO), b00 = (SBASE) + boff[0][0], b01 = (SBASE) + boff[0][1], b02 = (SBASE) + boff[0][2], b03 = (SBASE) + boff[0][3]; \
        const unsigned b10 = (SBASE) + boff[1][0], b11 = (SBASE) + boff[1][1], b12 = (SBASE) + boff[1][2], b13 = (SBASE) + boff[1][3]; \
        const bf16x8 f0 = CAT(C.l[0], C.h[0]), f1 = CAT(C.l[1], C.h[1]), f2 = CAT(C.l[2], C.h[2]), f3 = CAT(C.l[3], C.h[3]); \
        SB; M1(0, 0, f0, C.a[0]); SB; if (NX) { RTR(D.l[0], b00, BO); RTR(D.h[0], b10, BO); } \
        SB; M1(0, 1, f1, C.a[0]); SB; if (NX) { R128(D.a[0], aa, 0); } \
        SB; M1(0, 2, f2, C.a[0]); SB; if (NX) { RTR(D.l[1], b01, BO); RTR(D.h[1], b11, BO); } \
        SB; M1(0, 3, f3, C.a[0]); SB; if (NX) { R128(D.a[1], aa, 2048); } \
        SB; M1(1, 0, f0, C.a[1]); SB; if (NX) { RTR(D.l[2], b02, BO); RTR(D.h[2], b12, BO); } \
        SB; M1(1, 1, f1, C.a[1]); SB; if (NX) { R128(D.a[2], aa, 4096); } \
        SB; M1(1, 2, f2, C.a[1]); SB; if (NX) { RTR(D.l[3], b03, BO); RTR(D.h[3], b13, BO); } \
        SB; M1(1, 3, f3, C.a[1]); SB; if (NX) { R128(D.a[3], aa, 6144); } \
        SB; M1(2, 0, f0, C.a[2]); M1(2, 1, f1, C.a[2]); M1(2, 2, f2, C.a[2]); M1(2, 3, f3, C.a[2]); \
        M1(3, 0, f0, C.a[3]); M1(3, 1, f1, C.a[3]); M1(3, 2, f2, C.a[3]); M1(3, 3, f3, C.a[3]); SB; } while (0)
    __builtin_amdgcn_s_barrier();
    asm volatile("" ::: "memory");
    {
        const unsigned sb = smem_base;
#pragma unroll
        for (int nb = 0; nb < 4; ++nb) { RTR(X.l[nb], sb + boff[0][nb], 0); RTR(X.h[nb], sb + boff[1][nb], 0); }
        const unsigned aa = sb + aoff0;
        R128(X.a[0], aa, 0); R128(X.a[1], aa, 2048); R128(X.a[2], aa, 4096); R128(X.a[3], aa, 6144);
    }
    WL(X);
    int slot = 0;
    while (c < NST) {
        __builtin_amdgcn_s_barrier();
        asm volatile("" ::: "memory");
        int relv = NST;
        if (rel < NST) asm volatile("ds_read_b32 %0, %1\n\ts_waitcnt lgkmcnt(0)" : "=v"(relv) : "v"(relw + 4 * (it & 1)) : "memory");
        MACHINE_STEP(act, can);
        if (act) {
            const unsigned sb = smem_base + slot * SLOT_BYTES;
            const int nslot = slot == NSLOT - 1 ? 0 : slot + 1;
            const unsigned sn = smem_base + nslot * SLOT_BYTES;
            HALF(X, Y, sb, aoff1, 8192, true);
            WL(Y);
            HALF(Y, X, sn, aoff0, 0, c + 1 < NST);
            WL(X);
            slot = nslot;
        }
        MACHINE_NEXT(act, can);
    }
#undef HALF
#undef M1
#undef WL
#undef CAT
#undef RTR
#undef R128
#undef SB
    if (wave == 0) STAMP(10);

    f32x4 keep[2][4];
    {
        char* xb = myws + WS_X + ((size_t)g * 4 + wave) * 8192 + lane * 16;
#pragma unroll
        for (int a = 0; a < 2; ++a)
#pragma unroll
            for (int b = 0; b < 4; ++b) {
                store16_agent(xb + (a * 4 + b) * 1024, h ? acc[a][b] : acc[2 + a][b]);
                keep[a][b] = h ? acc[2 + a][b] : acc[a][b];
            }
        asm volatile("s_waitcnt vmcnt(0)" ::: "memory");
    }
    __syncthreads();
    if (tid == 0)
        __hip_atomic_store((unsigned*)(myws + WS_XFLAG + g * 64), call, __ATOMIC_RELAXED, __HIP_MEMORY_SCOPE_AGENT);
    float* red = (float*)(smem + 16384);
    {
        const int gp = g ^ 8;
        while (ld_agent32(myws + WS_XFLAG + gp * 64) != call) {
            if (give_up(tmo, call, t0)) break;
            __builtin_amdgcn_s_sleep(1);
        }
        const char* xb = myws + WS_X + ((size_t)gp * 4 + wave) * 8192 + lane * 16;
        f32x4 r0, r1, r2, r3, r4, r5, r6, r7;
        asm volatile("global_load_dwordx4 %0, %8, off sc1\n\t"
                     "global_load_dwordx4 %1, %8, off offset:1024 sc1\n\t"
                     "global_load_dwordx4 %2, %8, off offset:2048 sc1\n\t"
                     "global_load_dwordx4 %3, %8, off offset:3072 sc1\n\t"
                     "global_load_dwordx4 %4, %9, off sc1\n\t"
                     "global_load_dwordx4 %5, %9, off offset:1024 sc1\n\t"
                     "global_load_dwordx4 %6, %9, off offset:2048 sc1\n\t"
                     "global_load_dwordx4 %7, %9, off offset:3072 sc1\n\t"
                     "s_waitcnt vmcnt(0)"
                     : "=&v"(r0), "=&v"(r1), "=&v"(r2), "=&v"(r3), "=&v"(r4), "=&v"(r5), "=&v"(r6), "=&v"(r7)
                     : "v"(xb), "v"(xb + 4096) : "memory");
        keep[0][0] += r0; keep[0][1] += r1; keep[0][2] += r2; keep[0][3] += r3;
        keep[1][0] += r4; keep[1][1] += r5; keep[1][2] += r6; keep[1][3] += r7;
        float m = 0.f;
#pragma unroll
        for (int a = 0; a < 2; ++a)
#pragma unroll
            for (int b = 0; b < 4; ++b)
#pragma unroll
                for (int r = 0; r < 4; ++r) m = fmaxf(m, fabsf(keep[a][b][r]));
#pragma unroll
        for (int o = 32; o > 0; o >>= 1) m = fmaxf(m, __shfl_xor(m, o));
        if (lane == 0) red[wave] = m;
    }
    __syncthreads();
    if (wave == 0) STAMP(11);

    if (wave == 0) {
        const float wgm = fmaxf(fmaxf(red[0], red[1]), fmaxf(red[2], red[3]));
        const unsigned long long tag = (unsigned long long)call << 32;
        unsigned long long* loc = (unsigned long long*)(myws + WS_LOCAL);
        if (lane == 0)
            __hip_atomic_store(loc + g, tag | __float_as_uint(wgm), __ATOMIC_RELAXED, __HIP_MEMORY_SCOPE_AGENT);
        if (idx == 0) {
            unsigned long long v;
            for (;;) {
                v = ld_agent64(loc + grp + 8 * (lane & 31));
                if (__all((unsigned)(v >> 32) == call)) break;
                if (give_up(tmo, call, t0)) break;
                __builtin_amdgcn_s_sleep(1);
            }
            unsigned gm = (unsigned)v;
#pragma unroll
            for (int o = 32; o > 0; o >>= 1) { unsigned x = __shfl_xor(gm, o); gm = x > gm ? x : gm; }
            __hip_atomic_store((unsigned long long*)(p.ws[lane >> 3] + WS_GRAN) + (lane & 7) * 64 + rank * 8 + grp, tag | gm,
                               __ATOMIC_RELAXED, __HIP_MEMORY_SCOPE_SYSTEM);
        }
        unsigned long long v;
        const unsigned long long* gran = (const unsigned long long*)(myws + WS_GRAN) + grp * 64;
        for (;;) {
            v = ld_sys64(gran + lane);
            if (__all((unsigned)(v >> 32) == call)) break;
            if (give_up(tmo, call, t0)) break;
            __builtin_amdgcn_s_sleep(1);
        }
        unsigned gm = (unsigned)v;
#pragma unroll
        for (int o = 32; o > 0; o >>= 1) { unsigned x = __shfl_xor(gm, o); gm = x > gm ? x : gm; }
        if (lane == 0) {
            red[4] = __uint_as_float(gm);
            ((unsigned*)red)[5] = (ld_agent32(tmo) == call) ? 1u : 0u;
            if (g == 0) __hip_atomic_store((unsigned*)(myws + WS_EPOCH), call, __ATOMIC_RELAXED, __HIP_MEMORY_SCOPE_AGENT);
        }
    }
    __syncthreads();
    if (wave == 0) STAMP(12);
    const float amax = red[4];
    const bool bad = ((unsigned*)red)[5] != 0u;
    const float inv = amax > 0.f ? 127.f / amax : 0.f;

    constexpr int OSTR = BN + 16;
    char* ot = smem;
#pragma unroll
    for (int mb = 0; mb < 2; ++mb)
#pragma unroll
        for (int nb = 0; nb < 4; ++nb) {
            unsigned pk = 0;
#pragma unroll
            for (int r = 0; r < 4; ++r) {
                float q = rintf(keep[mb][nb][r] * inv);
                q = fminf(fmaxf(q, -127.f), 127.f);
                int qi = bad ? -128 : (int)q;
                pk |= ((unsigned)qi & 0xffu) << (8 * r);
            }
            const int row = wm * 32 + mb * 16 + fr, col = wn * 64 + nb * 16 + fq * 4;
            *(unsigned*)(ot + row * OSTR + col) = pk;
        }
    __syncthreads();
#pragma unroll
    for (int i = 0; i < 2; ++i) {
        const int row = (tid >> 3) + 32 * i, c16 = tid & 7;
        const u32x4 v = *(const u32x4*)(ot + row * OSTR + c16 * 16);
        *(u32x4*)(p.out + (size_t)(mt * BM + i * 64 + h * 32 + (tid >> 3)) * NN + nt * BN + c16 * 16) = v;
    }
    if (wave == 0) STAMP(13);
}

}

extern "C" void kernel_launch(int rank, int nranks, ncclComm_t comm,
                              hipStream_t stream,
                              void* const* d_in, const int* in_sizes, int n_in,
                              void* d_out, int out_size,
                              const int* in_dtypes, int out_dtype,
                              void* const* d_out_all,
                              void* const* d_ws_all, size_t ws_bytes) {
    if (nranks != NR || n_in < 2 || ws_bytes < WS_END || (unsigned)rank >= (unsigned)NR) {
        fprintf(stderr, "unsupported configuration\n");
        return;
    }
    Params p;
    p.x = (const char*)d_in[0];
    p.w = (const char*)d_in[1];
    p.out = (signed char*)d_out;
    for (int i = 0; i < NR; ++i) p.ws[i] = (char*)d_ws_all[i];
    p.rank = rank;
    hipLaunchKernelGGL(a2a_gemm_q8, dim3(N_WG), dim3(NT), 0, stream, p);
    HIP_CHECK(hipGetLastError());
}
```

```cpp
#include <hip/hip_runtime.h>
#include <rccl/rccl.h>
#include <cstdio>
#include <cstdint>

#define HIP_CHECK(expr) do { hipError_t _e = (expr); if (_e != hipSuccess) { \
    fprintf(stderr, "HIP error %s at %s:%d\n", hipGetErrorString(_e), __FILE__, __LINE__); \
    return; } } while (0)

namespace {

typedef short bf16x8 __attribute__((ext_vector_type(8)));
typedef short s16x4 __attribute__((ext_vector_type(4)));
typedef float f32x4 __attribute__((ext_vector_type(4)));
typedef unsigned u32x4 __attribute__((ext_vector_type(4)));

constexpr int NR = 8;
constexpr int MLOC = 512;
constexpr int KB = 512;
constexpr int NN = 4096;
constexpr int CH = 64;
constexpr int NCH = KB / CH;
constexpr int BM = 128, BN = 128;
constexpr int NST = NR * NCH / 2;
constexpr int NOWN = NCH / 2;
constexpr int NSLOT = 4;
constexpr int A_BYTES = BM * CH * 2;
constexpr int B_BYTES = CH * BN * 2;
constexpr int SLOT_BYTES = A_BYTES + B_BYTES;
constexpr int N_WG = 256;
constexpr int NRG = MLOC / 16;
constexpr int NT = 640;
#define POLL_LOW 3
#define PUSH_WIN 1

constexpr size_t WS_A = 0;
constexpr size_t WS_X = (size_t)NR * KB * MLOC * 2;
constexpr size_t WS_FLAG = WS_X + (size_t)N_WG * 32768;
constexpr size_t WS_XFLAG = WS_FLAG + NR * NCH * 4 * 128;
constexpr size_t WS_GRAN = WS_XFLAG + N_WG * 64;
constexpr size_t WS_LOCAL = WS_GRAN + 8 * 512;
constexpr size_t WS_EPOCH = WS_LOCAL + 2048;
constexpr size_t WS_TMO = WS_EPOCH + 128;
constexpr size_t WS_END = WS_TMO + 128;

constexpr unsigned long long DEADLINE_TICKS = 50000000ull;

struct Params {
    const char* x;
    const char* w;
    signed char* out;
    char* ws[NR];
    int rank;
};

__device__ __forceinline__ unsigned ld_agent32(const void* p) {
    return __hip_atomic_load((const unsigned*)p, __ATOMIC_RELAXED, __HIP_MEMORY_SCOPE_AGENT);
}
__device__ __forceinline__ unsigned long long ld_sys64(const void* p) {
    return __hip_atomic_load((const unsigned long long*)p, __ATOMIC_RELAXED, __HIP_MEMORY_SCOPE_SYSTEM);
}
__device__ __forceinline__ unsigned long long ld_agent64(const void* p) {
    return __hip_atomic_load((const unsigned long long*)p, __ATOMIC_RELAXED, __HIP_MEMORY_SCOPE_AGENT);
}

__device__ __forceinline__ bool give_up(unsigned* tmo, unsigned call, unsigned long long t0) {
    if (ld_agent32(tmo) == call || __builtin_amdgcn_s_memrealtime() - t0 > DEADLINE_TICKS) {
        __hip_atomic_store(tmo, call, __ATOMIC_RELAXED, __HIP_MEMORY_SCOPE_AGENT);
        return true;
    }
    return false;
}

__device__ __forceinline__ void store16_sys(void* p, u32x4 v) {
    asm volatile("global_store_dwordx4 %0, %1, off sc0 sc1\n\ts_nop 1" :: "v"(p), "v"(v) : "memory");
}
__device__ __forceinline__ void store16_agent(void* p, f32x4 v) {
    asm volatile("global_store_dwordx4 %0, %1, off sc1\n\ts_nop 1" :: "v"(p), "v"(v) : "memory");
}
__device__ __forceinline__ void dma16(unsigned voff, const char* sbase, unsigned lds) {
    asm volatile("s_mov_b32 m0, %2\n\ts_nop 0\n\tglobal_load_lds_dwordx4 %0, %1" :: "v"(voff), "s"(sbase), "s"(lds) : "memory");
}
__device__ __forceinline__ unsigned vm_outstanding() {
    unsigned s;
    asm volatile("s_getreg_b32 %0, hwreg(HW_REG_IB_STS)" : "=s"(s) :: "memory");
    return (s & 15u) | ((s >> 18) & 48u);
}

#define STAMP(i) do { } while (0)

__device__ __forceinline__ int swzB(int row) { return ((row & 3) << 2) | ((row >> 2) & 3); }

struct Frag { bf16x8 a[4]; s16x4 l[4], h[4]; };

__global__ void __launch_bounds__(NT, 3) a2a_gemm_q8(Params p) {
    __shared__ __attribute__((aligned(16))) char smem[NSLOT * SLOT_BYTES + 64 + 128 + 1024];
    const int tid = threadIdx.x, lane = tid & 63;
    const int wave = __builtin_amdgcn_readfirstlane(tid >> 6);
    const int rank = p.rank;
    char* myws = p.ws[rank];
    unsigned* tmo = (unsigned*)(myws + WS_TMO);
    const unsigned long long t0 = __builtin_amdgcn_s_memrealtime();

    const int g = blockIdx.x;
    const int grp = g & 7, idx = g >> 3;
    const int h = idx & 1, nt = grp * 4 + ((idx >> 1) & 3), mt = idx >> 3;
    const unsigned smem_base = (unsigned)(size_t)smem;
    const unsigned relw = smem_base + NSLOT * SLOT_BYTES;
    int c = 0, iss = NSLOT - 1, rel = NOWN, it = 0;
#define MACHINE_STEP(ACTIVE, CAN) \
        const bool ACTIVE = c + 1 < iss || c + 1 >= NST; \
        const int CAN = max(0, min(min(rel, c + NSLOT), NST) - iss);
#define MACHINE_NEXT(ACTIVE, CAN) do { rel = __builtin_amdgcn_readfirstlane(relv); iss += CAN; c += ACTIVE ? 1 : 0; ++it; } while (0)

    if (wave == 8) {
        const int j = g / NRG, rg = g % NRG;
        const int dst = (rank + 1 + j) & (NR - 1);
        const bool active = g < 7 * NRG && (unsigned)dst < (unsigned)NR;
        char* dws = p.ws[active ? dst : rank];
        const char* src = p.x + (size_t)((active ? dst : rank) * MLOC + rg * 16 + (lane >> 3)) * (KB * 2) + (lane & 7) * 16;
        char* dbase = dws + WS_A + ((size_t)rank * NCH * MLOC + rg * 16) * (CH * 2) + lane * 16;
        unsigned* flag = (unsigned*)(dws + WS_FLAG) + (rg >> 3) * 512 + rank * 8 + (rg & 7);
        STAMP(0);
        u32x4 v[NCH][2];
#pragma unroll
        for (int kc = 0; kc < NCH; ++kc)
#pragma unroll
            for (int q = 0; q < 2; ++q)
                asm volatile("global_load_dwordx4 %0, %1, off" : "=v"(v[kc][q]) : "v"(src + (size_t)q * 8 * (KB * 2) + kc * (CH * 2)) : "memory");
        unsigned epoch;
        asm volatile("global_load_dword %0, %1, off sc1" : "=v"(epoch) : "v"(myws + WS_EPOCH) : "memory");
        int niss = active ? 0 : NCH, npub = active ? 0 : NCH;
        unsigned nops = 0;
        unsigned long long endq = 0;
        asm volatile("s_waitcnt vmcnt(13)" : "+v"(v[0][0]), "+v"(v[0][1]), "+v"(v[1][0]), "+v"(v[1][1]) :: "memory");
        if (active) {
            store16_sys(dbase, v[0][0]); store16_sys(dbase + 1024, v[0][1]);
            store16_sys(dbase + (size_t)MLOC * (CH * 2), v[1][0]); store16_sys(dbase + (size_t)MLOC * (CH * 2) + 1024, v[1][1]);
            nops = 4; endq = 2ull | (4ull << 8); niss = 2;
        }
        asm volatile("s_waitcnt vmcnt(4)"
                     : "+v"(v[2][0]), "+v"(v[2][1]), "+v"(v[3][0]), "+v"(v[3][1]), "+v"(v[4][0]), "+v"(v[4][1]), "+v"(v[5][0]), "+v"(v[5][1]),
                       "+v"(v[6][0]), "+v"(v[6][1]), "+v"(v[7][0]), "+v"(v[7][1]), "+v"(epoch) :: "memory");
        const unsigned call = __builtin_amdgcn_readfirstlane(epoch) + 1u;
        auto progress = [&]() {
            while (npub < niss && vm_outstanding() <= nops - (unsigned)((endq >> (8 * npub)) & 0xffull)) {
                if (lane == 0) __hip_atomic_store(flag + (npub & 1) * 256 + (npub >> 1) * 64, call, __ATOMIC_RELAXED, __HIP_MEMORY_SCOPE_SYSTEM);
                asm volatile("" ::: "memory");
                ++npub; ++nops;
            }
#pragma unroll
            for (int kc = 2; kc < NCH; ++kc)
                if (niss == kc && kc < npub + PUSH_WIN) {
                    store16_sys(dbase + (size_t)kc * MLOC * (CH * 2), v[kc][0]);
                    store16_sys(dbase + (size_t)kc * MLOC * (CH * 2) + 1024, v[kc][1]);
                    nops += 2; endq |= (unsigned long long)nops << (8 * kc); ++niss;
                }
        };
        progress();
        __builtin_amdgcn_s_barrier();
        while (c < NST && npub < NCH) {
            progress();
            __builtin_amdgcn_s_barrier();
            asm volatile("" ::: "memory");
            int relv = NST;
            if (rel < NST) asm volatile("ds_read_b32 %0, %1\n\ts_waitcnt lgkmcnt(0)" : "=v"(relv) : "v"(relw + 4 * (it & 1)) : "memory");
            MACHINE_STEP(act, can);
            MACHINE_NEXT(act, can);
        }
        while (npub < NCH) { progress(); __builtin_amdgcn_s_sleep(1); }
        return;
    }

    if (wave == 9) {
        const unsigned call = ld_agent32(myws + WS_EPOCH) + 1u;
        asm volatile("s_waitcnt vmcnt(0)" ::: "memory");
        const char* fbase = myws + WS_FLAG + (mt * 2 + h) * 1024;
        unsigned todo = ~(0x01010101u << rank);
        int prel = NOWN;
        bool pending = false;
        const unsigned ordw = relw + 64, pbuf = relw + 64 + 128;
        const unsigned pvo = lane * 16;
#define POLL_ISSUE() do { \
            asm volatile("s_mov_b32 m0, %2\n\ts_nop 0\n\tglobal_load_lds_dwordx4 %0, %1 sc0 sc1" \
                         :: "v"(pvo), "s"(fbase), "s"(pbuf) : "memory"); \
            pending = true; } while (0)
        POLL_ISSUE();
        __builtin_amdgcn_s_barrier();
        while (c < NST && rel < NST) {
            if (pending && vm_outstanding() == 0) {
                u32x4 f, f2;
                asm volatile("ds_read_b128 %0, %2\n\tds_read_b128 %1, %2 offset:16\n\ts_waitcnt lgkmcnt(0)"
                             : "=&v"(f), "=&v"(f2) : "v"(pbuf + (lane & 31) * 32) : "memory");
                const unsigned lo = min(min(min(f[0], f[1]), min(f[2], f[3])), min(min(f2[0], f2[1]), min(f2[2], f2[3])));
                unsigned fresh = (unsigned)__ballot(lo >= call) & todo;
                pending = false;
                todo &= ~fresh;
                while (fresh) {
                    const int b = __builtin_ctz(fresh);
                    asm volatile("ds_write_b32 %0, %1" :: "v"(ordw + 4 * prel), "v"(b) : "memory");
                    fresh &= fresh - 1u; ++prel;
                }
            }
            if (prel < NST && !pending && prel - iss < POLL_LOW) {
                if ((it & 63) == 63 && give_up(tmo, call, t0)) {
                    for (; prel < NST; ++prel) asm volatile("ds_write_b32 %0, %1" :: "v"(ordw + 4 * prel), "v"(rank) : "memory");
                } else POLL_ISSUE();
            }
            asm volatile("ds_write_b32 %0, %1\n\ts_waitcnt lgkmcnt(0)" :: "v"(relw + 4 * (it & 1)), "v"(prel) : "memory");
            __builtin_amdgcn_s_barrier();
            asm volatile("" ::: "memory");
            const int relv = prel;
            MACHINE_STEP(act, can);
            MACHINE_NEXT(act, can);
        }
#undef POLL_ISSUE
        asm volatile("s_waitcnt vmcnt(0)" ::: "memory");
        return;
    }

    if (wave >= 4) {
        const int lw = wave - 4;
        unsigned avo_own[4], avo_rem[4], bvo[4];
#pragma unroll
        for (int i = 0; i < 4; ++i) {
            const int row = lw * 32 + i * 8 + (lane >> 3);
            const int ch = ((lane & 7) ^ ((row >> 1) & 7)) << 4;
            avo_own[i] = row * (KB * 2) + ch;
            avo_rem[i] = row * (CH * 2) + ch;
            const int brow = lw * 16 + i * 4 + (lane >> 4);
            bvo[i] = brow * (NN * 2) + (((lane & 15) ^ swzB(brow)) << 4);
        }
        const char* a_own = p.x + (size_t)(rank * MLOC + mt * BM) * (KB * 2) + h * (CH * 2);
        const char* a_rem = myws + WS_A + (size_t)(h * MLOC + mt * BM) * (CH * 2);
        const char* b0 = p.w + (size_t)h * CH * (NN * 2) + nt * (BN * 2);
        int islot = 0;
        auto issue = [&](int s) {
            const unsigned buf = smem_base + islot * SLOT_BYTES + lw * 4096;
            if (s < NOWN) {
                const char* ab = a_own + s * (2 * CH * 2);
                const char* bb = b0 + (size_t)(rank * KB + s * 2 * CH) * (NN * 2);
#pragma unroll
                for (int i = 0; i < 4; ++i) { dma16(avo_own[i], ab, buf + i * 1024); dma16(bvo[i], bb, buf + A_BYTES + i * 1024); }
            } else {
                int o;
                asm volatile("ds_read_b32 %0, %1\n\ts_waitcnt lgkmcnt(0)" : "=v"(o) : "v"(relw + 64 + 4 * s) : "memory");
                o = __builtin_amdgcn_readfirstlane(o);
                const int ir = (o >> 3) & 3, src = o & (NR - 1);
                const char* ab = a_rem + (size_t)(src * NCH + 2 * ir) * MLOC * (CH * 2);
                const char* bb = b0 + (size_t)(src * KB + ir * 2 * CH) * (NN * 2);
#pragma unroll
                for (int i = 0; i < 4; ++i) { dma16(avo_rem[i], ab, buf + i * 1024); dma16(bvo[i], bb, buf + A_BYTES + i * 1024); }
            }
            islot = islot == NSLOT - 1 ? 0 : islot + 1;
        };
#pragma unroll
        for (int s = 0; s < NSLOT - 1; ++s) issue(s);
        asm volatile("s_waitcnt vmcnt(%0)" :: "n"(8 * (NSLOT - 2)) : "memory");
        __builtin_amdgcn_s_barrier();
        while (c < NST) {
            if (iss >= c + 4) asm volatile("s_waitcnt vmcnt(16)" ::: "memory");
            else if (iss == c + 3) asm volatile("s_waitcnt vmcnt(8)" ::: "memory");
            else asm volatile("s_waitcnt vmcnt(0)" ::: "memory");
            __builtin_amdgcn_s_barrier();
            asm volatile("" ::: "memory");
            MACHINE_STEP(act, can);
            for (int k = 0; k < can; ++k) issue(iss + k);
            int relv = NST;
            if (rel < NST) asm volatile("ds_read_b32 %0, %1\n\ts_waitcnt lgkmcnt(0)" : "=v"(relv) : "v"(relw + 4 * (it & 1)) : "memory");
            MACHINE_NEXT(act, can);
        }
        return;
    }

    STAMP(0);
    const unsigned call = ld_agent32(myws + WS_EPOCH) + 1u;
    const int wm = wave >> 1, wn = wave & 1;
    f32x4 acc[4][4];
#pragma unroll
    for (int a = 0; a < 4; ++a)
#pragma unroll
        for (int b = 0; b < 4; ++b) acc[a][b] = (f32x4){0.f, 0.f, 0.f, 0.f};
    const int fr = lane & 15, fq = lane >> 4;
    const int aoff0 = (wm * 64 + fr) * 128 + (((0 + fq) ^ ((fr >> 1) & 7)) << 4);
    const int aoff1 = (wm * 64 + fr) * 128 + (((4 + fq) ^ ((fr >> 1) & 7)) << 4);
    int boff[2][4];
#pragma unroll
    for (int t = 0; t < 2; ++t) {
        const int row = 8 * fq + 4 * t + ((lane & 15) >> 2);
#pragma unroll
        for (int nb = 0; nb < 4; ++nb)
            boff[t][nb] = A_BYTES + row * 256 + 8 * (lane & 1) + (((wn * 8 + 2 * nb + ((lane & 3) >> 1)) ^ swzB(row)) << 4);
    }
    Frag X, Y;
#define SB __builtin_amdgcn_sched_barrier(0)
#define R128(D, ADDR, OFF) asm volatile("ds_read_b128 %0, %1 offset:" #OFF : "=v"(D) : "v"(ADDR) : "memory")
#define RTR(D, ADDR, OFF) asm volatile("ds_read_b64_tr_b16 %0, %1 offset:" #OFF : "=v"(D) : "v"(ADDR) : "memory")
#define CAT(lo, hi) __builtin_shufflevector(lo, hi, 0, 1, 2, 3, 4, 5, 6, 7)
#define WL(F) asm volatile("s_waitcnt lgkmcnt(0)" : "+v"(F.a[0]), "+v"(F.a[1]), "+v"(F.a[2]), "+v"(F.a[3]), \
        "+v"(F.l[0]), "+v"(F.h[0]), "+v"(F.l[1]), "+v"(F.h[1]), "+v"(F.l[2]), "+v"(F.h[2]), "+v"(F.l[3]), "+v"(F.h[3]) :: "memory")
#define M1(I, J, F, A) acc[I][J] = __builtin_amdgcn_mfma_f32_16x16x32_bf16(F, A, acc[I][J], 0, 0, 0)
#define HALF(C, D, SBASE, AO, BO, NX) do { \
        const unsigned aa = (SBASE) + (AO), b00 = (SBASE) + boff[0][0], b01 = (SBASE) + boff[0][1], b02 = (SBASE) + boff[0][2], b03 = (SBASE) + boff[0][3]; \
        const unsigned b10 = (SBASE) + boff[1][0], b11 = (SBASE) + boff[1][1], b12 = (SBASE) + boff[1][2], b13 = (SBASE) + boff[1][3]; \
        const bf16x8 f0 = CAT(C.l[0], C.h[0]), f1 = CAT(C.l[1], C.h[1]), f2 = CAT(C.l[2], C.h[2]), f3 = CAT(C.l[3], C.h[3]); \
        SB; M1(0, 0, f0, C.a[0]); SB; if (NX) { RTR(D.l[0], b00, BO); RTR(D.h[0], b10, BO); } \
        SB; M1(0, 1, f1, C.a[0]); SB; if (NX) { R128(D.a[0], aa, 0); } \
        SB; M1(0, 2, f2, C.a[0]); SB; if (NX) { RTR(D.l[1], b01, BO); RTR(D.h[1], b11, BO); } \
        SB; M1(0, 3, f3, C.a[0]); SB; if (NX) { R128(D.a[1], aa, 2048); } \
        SB; M1(1, 0, f0, C.a[1]); SB; if (NX) { RTR(D.l[2], b02, BO); RTR(D.h[2], b12, BO); } \
        SB; M1(1, 1, f1, C.a[1]); SB; if (NX) { R128(D.a[2], aa, 4096); } \
        SB; M1(1, 2, f2, C.a[1]); SB; if (NX) { RTR(D.l[3], b03, BO); RTR(D.h[3], b13, BO); } \
        SB; M1(1, 3, f3, C.a[1]); SB; if (NX) { R128(D.a[3], aa, 6144); } \
        SB; M1(2, 0, f0, C.a[2]); M1(2, 1, f1, C.a[2]); M1(2, 2, f2, C.a[2]); M1(2, 3, f3, C.a[2]); \
        M1(3, 0, f0, C.a[3]); M1(3, 1, f1, C.a[3]); M1(3, 2, f2, C.a[3]); M1(3, 3, f3, C.a[3]); SB; } while (0)
    __builtin_amdgcn_s_barrier();
    asm volatile("" ::: "memory");
    {
        const unsigned sb = smem_base;
#pragma unroll
        for (int nb = 0; nb < 4; ++nb) { RTR(X.l[nb], sb + boff[0][nb], 0); RTR(X.h[nb], sb + boff[1][nb], 0); }
        const unsigned aa = sb + aoff0;
        R128(X.a[0], aa, 0); R128(X.a[1], aa, 2048); R128(X.a[2], aa, 4096); R128(X.a[3], aa, 6144);
    }
    WL(X);
    int slot = 0;
    while (c < NST) {
        __builtin_amdgcn_s_barrier();
        asm volatile("" ::: "memory");
        int relv = NST;
        if (rel < NST) asm volatile("ds_read_b32 %0, %1\n\ts_waitcnt lgkmcnt(0)" : "=v"(relv) : "v"(relw + 4 * (it & 1)) : "memory");
        MACHINE_STEP(act, can);
        if (act) {
            const unsigned sb = smem_base + slot * SLOT_BYTES;
            const int nslot = slot == NSLOT - 1 ? 0 : slot + 1;
            const unsigned sn = smem_base + nslot * SLOT_BYTES;
            HALF(X, Y, sb, aoff1, 8192, true);
            WL(Y);
            HALF(Y, X, sn, aoff0, 0, true);
            WL(X);
            slot = nslot;
        }
        MACHINE_NEXT(act, can);
    }
#undef HALF
#undef M1
#undef WL
#undef CAT
#undef RTR
#undef R128
#undef SB
    if (wave == 0) STAMP(10);

    f32x4 keep[2][4];
    {
        char* xb = myws + WS_X + ((size_t)g * 4 + wave) * 8192 + lane * 16;
#pragma unroll
        for (int a = 0; a < 2; ++a)
#pragma unroll
            for (int b = 0; b < 4; ++b) {
                store16_agent(xb + (a * 4 + b) * 1024, h ? acc[a][b] : acc[2 + a][b]);
                keep[a][b] = h ? acc[2 + a][b] : acc[a][b];
            }
        asm volatile("s_waitcnt vmcnt(0)" ::: "memory");
    }
    __syncthreads();
    if (tid == 0)
        __hip_atomic_store((unsigned*)(myws + WS_XFLAG + g * 64), call, __ATOMIC_RELAXED, __HIP_MEMORY_SCOPE_AGENT);
    float* red = (float*)(smem + 16384);
    {
        const int gp = g ^ 8;
        while (ld_agent32(myws + WS_XFLAG + gp * 64) != call) {
            if (give_up(tmo, call, t0)) break;
            __builtin_amdgcn_s_sleep(1);
        }
        const char* xb = myws + WS_X + ((size_t)gp * 4 + wave) * 8192 + lane * 16;
        f32x4 r0, r1, r2, r3, r4, r5, r6, r7;
        asm volatile("global_load_dwordx4 %0, %8, off sc1\n\t"
                     "global_load_dwordx4 %1, %8, off offset:1024 sc1\n\t"
                     "global_load_dwordx4 %2, %8, off offset:2048 sc1\n\t"
                     "global_load_dwordx4 %3, %8, off offset:3072 sc1\n\t"
                     "global_load_dwordx4 %4, %9, off sc1\n\t"
                     "global_load_dwordx4 %5, %9, off offset:1024 sc1\n\t"
                     "global_load_dwordx4 %6, %9, off offset:2048 sc1\n\t"
                     "global_load_dwordx4 %7, %9, off offset:3072 sc1\n\t"
                     "s_waitcnt vmcnt(0)"
                     : "=&v"(r0), "=&v"(r1), "=&v"(r2), "=&v"(r3), "=&v"(r4), "=&v"(r5), "=&v"(r6), "=&v"(r7)
                     : "v"(xb), "v"(xb + 4096) : "memory");
        keep[0][0] += r0; keep[0][1] += r1; keep[0][2] += r2; keep[0][3] += r3;
        keep[1][0] += r4; keep[1][1] += r5; keep[1][2] += r6; keep[1][3] += r7;
        float m = 0.f;
#pragma unroll
        for (int a = 0; a < 2; ++a)
#pragma unroll
            for (int b = 0; b < 4; ++b)
#pragma unroll
                for (int r = 0; r < 4; ++r) m = fmaxf(m, fabsf(keep[a][b][r]));
#pragma unroll
        for (int o = 32; o > 0; o >>= 1) m = fmaxf(m, __shfl_xor(m, o));
        if (lane == 0) red[wave] = m;
    }
    __syncthreads();
    if (wave == 0) STAMP(11);

    if (wave == 0) {
        const float wgm = fmaxf(fmaxf(red[0], red[1]), fmaxf(red[2], red[3]));
        const unsigned long long tag = (unsigned long long)call << 32;
        unsigned long long* loc = (unsigned long long*)(myws + WS_LOCAL);
        if (lane == 0)
            __hip_atomic_store(loc + g, tag | __float_as_uint(wgm), __ATOMIC_RELAXED, __HIP_MEMORY_SCOPE_AGENT);
        if (idx == 0) {
            unsigned long long v;
            for (;;) {
                v = ld_agent64(loc + grp + 8 * (lane & 31));
                if (__all((unsigned)(v >> 32) == call)) break;
                if (give_up(tmo, call, t0)) break;
                __builtin_amdgcn_s_sleep(1);
            }
            unsigned gm = (unsigned)v;
#pragma unroll
            for (int o = 32; o > 0; o >>= 1) { unsigned x = __shfl_xor(gm, o); gm = x > gm ? x : gm; }
            __hip_atomic_store((unsigned long long*)(p.ws[lane >> 3] + WS_GRAN) + (lane & 7) * 64 + rank * 8 + grp, tag | gm,
                               __ATOMIC_RELAXED, __HIP_MEMORY_SCOPE_SYSTEM);
        }
        unsigned long long v;
        const unsigned long long* gran = (const unsigned long long*)(myws + WS_GRAN) + grp * 64;
        for (;;) {
            v = ld_sys64(gran + lane);
            if (__all((unsigned)(v >> 32) == call)) break;
            if (give_up(tmo, call, t0)) break;
            __builtin_amdgcn_s_sleep(1);
        }
        unsigned gm = (unsigned)v;
#pragma unroll
        for (int o = 32; o > 0; o >>= 1) { unsigned x = __shfl_xor(gm, o); gm = x > gm ? x : gm; }
        if (lane == 0) {
            red[4] = __uint_as_float(gm);
            ((unsigned*)red)[5] = (ld_agent32(tmo) == call) ? 1u : 0u;
            if (g == 0) __hip_atomic_store((unsigned*)(myws + WS_EPOCH), call, __ATOMIC_RELAXED, __HIP_MEMORY_SCOPE_AGENT);
        }
    }
    __syncthreads();
    if (wave == 0) STAMP(12);
    const float amax = red[4];
    const bool bad = ((unsigned*)red)[5] != 0u;
    const float inv = amax > 0.f ? 127.f / amax : 0.f;

    constexpr int OSTR = BN + 16;
    char* ot = smem;
#pragma unroll
    for (int mb = 0; mb < 2; ++mb)
#pragma unroll
        for (int nb = 0; nb < 4; ++nb) {
            unsigned pk = 0;
#pragma unroll
            for (int r = 0; r < 4; ++r) {
                float q = rintf(keep[mb][nb][r] * inv);
                q = fminf(fmaxf(q, -127.f), 127.f);
                int qi = bad ? -128 : (int)q;
                pk |= ((unsigned)qi & 0xffu) << (8 * r);
            }
            const int row = wm * 32 + mb * 16 + fr, col = wn * 64 + nb * 16 + fq * 4;
            *(unsigned*)(ot + row * OSTR + col) = pk;
        }
    __syncthreads();
#pragma unroll
    for (int i = 0; i < 2; ++i) {
        const int row = (tid >> 3) + 32 * i, c16 = tid & 7;
        const u32x4 v = *(const u32x4*)(ot + row * OSTR + c16 * 16);
        *(u32x4*)(p.out + (size_t)(mt * BM + i * 64 + h * 32 + (tid >> 3)) * NN + nt * BN + c16 * 16) = v;
    }
    if (wave == 0) STAMP(13);
}

}

extern "C" void kernel_launch(int rank, int nranks, ncclComm_t comm,
                              hipStream_t stream,
                              void* const* d_in, const int* in_sizes, int n_in,
                              void* d_out, int out_size,
                              const int* in_dtypes, int out_dtype,
                              void* const* d_out_all,
                              void* const* d_ws_all, size_t ws_bytes) {
    if (nranks != NR || n_in < 2 || ws_bytes < WS_END || (unsigned)rank >= (unsigned)NR) {
        fprintf(stderr, "unsupported configuration\n");
        return;
    }
    Params p;
    p.x = (const char*)d_in[0];
    p.w = (const char*)d_in[1];
    p.out = (signed char*)d_out;
    for (int i = 0; i < NR; ++i) p.ws[i] = (char*)d_ws_all[i];
    p.rank = rank;
    hipLaunchKernelGGL(a2a_gemm_q8, dim3(N_WG), dim3(NT), 0, stream, p);
    HIP_CHECK(hipGetLastError());
}
```

```cpp
#include <hip/hip_runtime.h>
#include <rccl/rccl.h>
#include <cstdio>
#include <cstdint>

#define HIP_CHECK(expr) do { hipError_t _e = (expr); if (_e != hipSuccess) { \
    fprintf(stderr, "HIP error %s at %s:%d\n", hipGetErrorString(_e), __FILE__, __LINE__); \
    return; } } while (0)

namespace {

typedef short bf16x8 __attribute__((ext_vector_type(8)));
typedef short s16x4 __attribute__((ext_vector_type(4)));
typedef float f32x4 __attribute__((ext_vector_type(4)));
typedef unsigned u32x4 __attribute__((ext_vector_type(4)));

constexpr int NR = 8;
constexpr int MLOC = 512;
constexpr int KB = 512;
constexpr int NN = 4096;
constexpr int CH = 64;
constexpr int NCH = KB / CH;
constexpr int BM = 128, BN = 128;
constexpr int NST = NR * NCH / 2;
constexpr int NOWN = NCH / 2;
constexpr int NSLOT = 4;
constexpr int A_BYTES = BM * CH * 2;
constexpr int B_BYTES = CH * BN * 2;
constexpr int SLOT_BYTES = A_BYTES + B_BYTES;
constexpr int N_WG = 256;
constexpr int NRG = MLOC / 16;
constexpr int NT = 640;
#define POLL_LOW 3
#define PUSH_WIN 1

constexpr size_t WS_A = 0;
constexpr size_t WS_X = (size_t)NR * KB * MLOC * 2;
constexpr size_t WS_FLAG = WS_X + (size_t)N_WG * 32768;
constexpr size_t WS_XFLAG = WS_FLAG + NR * NCH * 4 * 128;
constexpr size_t WS_GRAN = WS_XFLAG + N_WG * 64;
constexpr size_t WS_LOCAL = WS_GRAN + 8 * 512;
constexpr size_t WS_EPOCH = WS_LOCAL + 2048;
constexpr size_t WS_TMO = WS_EPOCH + 128;
constexpr size_t WS_END = WS_TMO + 128;

constexpr unsigned long long DEADLINE_TICKS = 50000000ull;

struct Params {
    const char* x;
    const char* w;
    signed char* out;
    char* ws[NR];
    int rank;
};

__device__ __forceinline__ unsigned ld_agent32(const void* p) {
    return __hip_atomic_load((const unsigned*)p, __ATOMIC_RELAXED, __HIP_MEMORY_SCOPE_AGENT);
}
__device__ __forceinline__ unsigned long long ld_sys64(const void* p) {
    return __hip_atomic_load((const unsigned long long*)p, __ATOMIC_RELAXED, __HIP_MEMORY_SCOPE_SYSTEM);
}
__device__ __forceinline__ unsigned long long ld_agent64(const void* p) {
    return __hip_atomic_load((const unsigned long long*)p, __ATOMIC_RELAXED, __HIP_MEMORY_SCOPE_AGENT);
}

__device__ __forceinline__ bool give_up(unsigned* tmo, unsigned call, unsigned long long t0) {
    if (ld_agent32(tmo) == call || __builtin_amdgcn_s_memrealtime() - t0 > DEADLINE_TICKS) {
        __hip_atomic_store(tmo, call, __ATOMIC_RELAXED, __HIP_MEMORY_SCOPE_AGENT);
        return true;
    }
    return false;
}

__device__ __forceinline__ void store16_sys(void* p, u32x4 v) {
    asm volatile("global_store_dwordx4 %0, %1, off sc0 sc1\n\ts_nop 1" :: "v"(p), "v"(v) : "memory");
}
__device__ __forceinline__ void store16_agent(void* p, f32x4 v) {
    asm volatile("global_store_dwordx4 %0, %1, off sc1\n\ts_nop 1" :: "v"(p), "v"(v) : "memory");
}
__device__ __forceinline__ void dma16(unsigned voff, const char* sbase, unsigned lds) {
    asm volatile("s_mov_b32 m0, %2\n\ts_nop 0\n\tglobal_load_lds_dwordx4 %0, %1" :: "v"(voff), "s"(sbase), "s"(lds) : "memory");
}
__device__ __forceinline__ unsigned vm_outstanding() {
    unsigned s;
    asm volatile("s_getreg_b32 %0, hwreg(HW_REG_IB_STS)" : "=s"(s) :: "memory");
    return (s & 15u) | ((s >> 18) & 48u);
}

#define STAMP(i) do { } while (0)

__device__ __forceinline__ int swzB(int row) { return ((row & 3) << 2) | ((row >> 2) & 3); }

struct Frag { bf16x8 a[4]; s16x4 l[4], h[4]; };

__global__ void __launch_bounds__(NT, 3) a2a_gemm_q8(Params p) {
    __shared__ __attribute__((aligned(16))) char smem[NSLOT * SLOT_BYTES + 64 + 128 + 1024];
    const int tid = threadIdx.x, lane = tid & 63;
    const int wave = __builtin_amdgcn_readfirstlane(tid >> 6);
    const int rank = p.rank;
    char* myws = p.ws[rank];
    unsigned* tmo = (unsigned*)(myws + WS_TMO);
    const unsigned long long t0 = __builtin_amdgcn_s_memrealtime();

    const int g = blockIdx.x;
    const int grp = g & 7, idx = g >> 3;
    const int h = idx & 1, nt = grp * 4 + ((idx >> 1) & 3), mt = idx >> 3;
    const unsigned smem_base = (unsigned)(size_t)smem;
    const unsigned relw = smem_base + NSLOT * SLOT_BYTES;
    int c = 0, iss = NSLOT - 1, rel = NOWN, it = 0;
#define MACHINE_STEP(ACTIVE, CAN) \
        const bool ACTIVE = c + 1 < iss || c + 1 >= NST; \
        const int CAN = max(0, min(min(rel, c + NSLOT), NST) - iss);
#define MACHINE_NEXT(ACTIVE, CAN) do { rel = __builtin_amdgcn_readfirstlane(relv); iss += CAN; c += ACTIVE ? 1 : 0; ++it; } while (0)

    if (wave == 8) {
        const int j = g / NRG, rg = g % NRG;
        const int dst = (rank + 1 + j) & (NR - 1);
        const bool active = g < 7 * NRG && (unsigned)dst < (unsigned)NR;
        char* dws = p.ws[active ? dst : rank];
        const char* src = p.x + (size_t)((active ? dst : rank) * MLOC + rg * 16 + (lane >> 3)) * (KB * 2) + (lane & 7) * 16;
        char* dbase = dws + WS_A + ((size_t)rank * NCH * MLOC + rg * 16) * (CH * 2) + lane * 16;
        unsigned* flag = (unsigned*)(dws + WS_FLAG) + (rg >> 3) * 512 + rank * 8 + (rg & 7);
        STAMP(0);
        u32x4 v[NCH][2];
#pragma unroll
        for (int kc = 0; kc < NCH; ++kc)
#pragma unroll
            for (int q = 0; q < 2; ++q)
                asm volatile("global_load_dwordx4 %0, %1, off" : "=v"(v[kc][q]) : "v"(src + (size_t)q * 8 * (KB * 2) + kc * (CH * 2)) : "memory");
        unsigned epoch;
        asm volatile("global_load_dword %0, %1, off sc1" : "=v"(epoch) : "v"(myws + WS_EPOCH) : "memory");
        int niss = active ? 0 : NCH, npub = active ? 0 : NCH;
        unsigned nops = 0;
        unsigned long long endq = 0;
        asm volatile("s_waitcnt vmcnt(15)" : "+v"(v[0][0]), "+v"(v[0][1]) :: "memory");
        if (active) {
            store16_sys(dbase, v[0][0]); store16_sys(dbase + 1024, v[0][1]);
            nops = 2; endq = 2ull; niss = 1;
        }
        asm volatile("s_waitcnt vmcnt(2)"
                     : "+v"(v[1][0]), "+v"(v[1][1]), "+v"(v[2][0]), "+v"(v[2][1]), "+v"(v[3][0]), "+v"(v[3][1]), "+v"(v[4][0]), "+v"(v[4][1]),
                       "+v"(v[5][0]), "+v"(v[5][1]), "+v"(v[6][0]), "+v"(v[6][1]), "+v"(v[7][0]), "+v"(v[7][1]), "+v"(epoch) :: "memory");
        const unsigned call = __builtin_amdgcn_readfirstlane(epoch) + 1u;
        auto progress = [&]() {
            while (npub < niss && vm_outstanding() <= nops - (unsigned)((endq >> (8 * npub)) & 0xffull)) {
                if (lane == 0) __hip_atomic_store(flag + ((npub + rank) & 1) * 256 + (npub >> 1) * 64, call, __ATOMIC_RELAXED, __HIP_MEMORY_SCOPE_SYSTEM);
                asm volatile("" ::: "memory");
                ++npub; ++nops;
            }
#pragma unroll
            for (int kc = 1; kc < NCH; ++kc)
                if (niss == kc && kc < npub + PUSH_WIN) {
                    store16_sys(dbase + (size_t)kc * MLOC * (CH * 2), v[kc][0]);
                    store16_sys(dbase + (size_t)kc * MLOC * (CH * 2) + 1024, v[kc][1]);
                    nops += 2; endq |= (unsigned long long)nops << (8 * kc); ++niss;
                }
        };
        progress();
        __builtin_amdgcn_s_barrier();
        while (c < NST && npub < NCH) {
            progress();
            __builtin_amdgcn_s_barrier();
            asm volatile("" ::: "memory");
            int relv = NST;
            if (rel < NST) asm volatile("ds_read_b32 %0, %1\n\ts_waitcnt lgkmcnt(0)" : "=v"(relv) : "v"(relw + 4 * (it & 1)) : "memory");
            MACHINE_STEP(act, can);
            MACHINE_NEXT(act, can);
        }
        while (npub < NCH) { progress(); __builtin_amdgcn_s_sleep(1); }
        return;
    }

    if (wave == 9) {
        const unsigned call = ld_agent32(myws + WS_EPOCH) + 1u;
        asm volatile("s_waitcnt vmcnt(0)" ::: "memory");
        const char* fbase = myws + WS_FLAG + (mt * 2 + h) * 1024;
        unsigned todo = ~(0x01010101u << rank);
        int prel = NOWN;
        bool pending = false;
        const unsigned ordw = relw + 64, pbuf = relw + 64 + 128;
        const unsigned pvo = lane * 16;
#define POLL_ISSUE() do { \
            asm volatile("s_mov_b32 m0, %2\n\ts_nop 0\n\tglobal_load_lds_dwordx4 %0, %1 sc0 sc1" \
                         :: "v"(pvo), "s"(fbase), "s"(pbuf) : "memory"); \
            pending = true; } while (0)
        POLL_ISSUE();
        __builtin_amdgcn_s_barrier();
        while (c < NST && rel < NST) {
            if (pending && vm_outstanding() == 0) {
                u32x4 f, f2;
                asm volatile("ds_read_b128 %0, %2\n\tds_read_b128 %1, %2 offset:16\n\ts_waitcnt lgkmcnt(0)"
                             : "=&v"(f), "=&v"(f2) : "v"(pbuf + (lane & 31) * 32) : "memory");
                const unsigned lo = min(min(min(f[0], f[1]), min(f[2], f[3])), min(min(f2[0], f2[1]), min(f2[2], f2[3])));
                unsigned fresh = (unsigned)__ballot(lo >= call) & todo;
                pending = false;
                todo &= ~fresh;
                while (fresh) {
                    const int b = __builtin_ctz(fresh);
                    asm volatile("ds_write_b32 %0, %1" :: "v"(ordw + 4 * prel), "v"(b) : "memory");
                    fresh &= fresh - 1u; ++prel;
                }
            }
            if (prel < NST && !pending && prel - iss < POLL_LOW) {
                if ((it & 63) == 63 && give_up(tmo, call, t0)) {
                    for (; prel < NST; ++prel) asm volatile("ds_write_b32 %0, %1" :: "v"(ordw + 4 * prel), "v"(rank) : "memory");
                } else POLL_ISSUE();
            }
            asm volatile("ds_write_b32 %0, %1\n\ts_waitcnt lgkmcnt(0)" :: "v"(relw + 4 * (it & 1)), "v"(prel) : "memory");
            __builtin_amdgcn_s_barrier();
            asm volatile("" ::: "memory");
            const int relv = prel;
            MACHINE_STEP(act, can);
            MACHINE_NEXT(act, can);
        }
#undef POLL_ISSUE
        asm volatile("s_waitcnt vmcnt(0)" ::: "memory");
        return;
    }

    if (wave >= 4) {
        const int lw = wave - 4;
        unsigned avo_own[4], avo_rem[4], bvo[4];
#pragma unroll
        for (int i = 0; i < 4; ++i) {
            const int row = lw * 32 + i * 8 + (lane >> 3);
            const int ch = ((lane & 7) ^ ((row >> 1) & 7)) << 4;
            avo_own[i] = row * (KB * 2) + ch;
            avo_rem[i] = row * (CH * 2) + ch;
            const int brow = lw * 16 + i * 4 + (lane >> 4);
            bvo[i] = brow * (NN * 2) + (((lane & 15) ^ swzB(brow)) << 4);
        }
        const char* a_own = p.x + (size_t)(rank * MLOC + mt * BM) * (KB * 2) + ((h + rank) & 1) * (CH * 2);
        const char* a_rem = myws + WS_A + (size_t)(mt * BM) * (CH * 2);
        const char* b0 = p.w + nt * (BN * 2);
        int islot = 0;
        auto issue = [&](int s) {
            const unsigned buf = smem_base + islot * SLOT_BYTES + lw * 4096;
            if (s < NOWN) {
                const char* ab = a_own + s * (2 * CH * 2);
                const char* bb = b0 + (size_t)(rank * KB + (2 * s + ((h + rank) & 1)) * CH) * (NN * 2);
#pragma unroll
                for (int i = 0; i < 4; ++i) { dma16(avo_own[i], ab, buf + i * 1024); dma16(bvo[i], bb, buf + A_BYTES + i * 1024); }
            } else {
                int o;
                asm volatile("ds_read_b32 %0, %1\n\ts_waitcnt lgkmcnt(0)" : "=v"(o) : "v"(relw + 64 + 4 * s) : "memory");
                o = __builtin_amdgcn_readfirstlane(o);
                const int ir = (o >> 3) & 3, src = o & (NR - 1);
                const int kc = 2 * ir + ((h + src) & 1);
                const char* ab = a_rem + (size_t)(src * NCH + kc) * MLOC * (CH * 2);
                const char* bb = b0 + (size_t)(src * KB + kc * CH) * (NN * 2);
#pragma unroll
                for (int i = 0; i < 4; ++i) { dma16(avo_rem[i], ab, buf + i * 1024); dma16(bvo[i], bb, buf + A_BYTES + i * 1024); }
            }
            islot = islot == NSLOT - 1 ? 0 : islot + 1;
        };
#pragma unroll
        for (int s = 0; s < NSLOT - 1; ++s) issue(s);
        asm volatile("s_waitcnt vmcnt(%0)" :: "n"(8 * (NSLOT - 2)) : "memory");
        __builtin_amdgcn_s_barrier();
        while (c < NST) {
            if (iss >= c + 4) asm volatile("s_waitcnt vmcnt(16)" ::: "memory");
            else if (iss == c + 3) asm volatile("s_waitcnt vmcnt(8)" ::: "memory");
            else asm volatile("s_waitcnt vmcnt(0)" ::: "memory");
            __builtin_amdgcn_s_barrier();
            asm volatile("" ::: "memory");
            MACHINE_STEP(act, can);
            for (int k = 0; k < can; ++k) issue(iss + k);
            int relv = NST;
            if (rel < NST) asm volatile("ds_read_b32 %0, %1\n\ts_waitcnt lgkmcnt(0)" : "=v"(relv) : "v"(relw + 4 * (it & 1)) : "memory");
            MACHINE_NEXT(act, can);
        }
        return;
    }

    STAMP(0);
    const unsigned call = ld_agent32(myws + WS_EPOCH) + 1u;
    const int wm = wave >> 1, wn = wave & 1;
    f32x4 acc[4][4];
#pragma unroll
    for (int a = 0; a < 4; ++a)
#pragma unroll
        for (int b = 0; b < 4; ++b) acc[a][b] = (f32x4){0.f, 0.f, 0.f, 0.f};
    const int fr = lane & 15, fq = lane >> 4;
    const int aoff0 = (wm * 64 + fr) * 128 + (((0 + fq) ^ ((fr >> 1) & 7)) << 4);
    const int aoff1 = (wm * 64 + fr) * 128 + (((4 + fq) ^ ((fr >> 1) & 7)) << 4);
    int boff[2][4];
#pragma unroll
    for (int t = 0; t < 2; ++t) {
        const int row = 8 * fq + 4 * t + ((lane & 15) >> 2);
#pragma unroll
        for (int nb = 0; nb < 4; ++nb)
            boff[t][nb] = A_BYTES + row * 256 + 8 * (lane & 1) + (((wn * 8 + 2 * nb + ((lane & 3) >> 1)) ^ swzB(row)) << 4);
    }
    Frag X, Y;
#define SB __builtin_amdgcn_sched_barrier(0)
#define R128(D, ADDR, OFF) asm volatile("ds_read_b128 %0, %1 offset:" #OFF : "=v"(D) : "v"(ADDR) : "memory")
#define RTR(D, ADDR, OFF) asm volatile("ds_read_b64_tr_b16 %0, %1 offset:" #OFF : "=v"(D) : "v"(ADDR) : "memory")
#define CAT(lo, hi) __builtin_shufflevector(lo, hi, 0, 1, 2, 3, 4, 5, 6, 7)
#define WL(F) asm volatile("s_waitcnt lgkmcnt(0)" : "+v"(F.a[0]), "+v"(F.a[1]), "+v"(F.a[2]), "+v"(F.a[3]), \
        "+v"(F.l[0]), "+v"(F.h[0]), "+v"(F.l[1]), "+v"(F.h[1]), "+v"(F.l[2]), "+v"(F.h[2]), "+v"(F.l[3]), "+v"(F.h[3]) :: "memory")
#define M1(I, J, F, A) acc[I][J] = __builtin_amdgcn_mfma_f32_16x16x32_bf16(F, A, acc[I][J], 0, 0, 0)
#define HALF(C, D, SBASE, AO, BO, NX) do { \
        const unsigned aa = (SBASE) + (AO), b00 = (SBASE) + boff[0][0], b01 = (SBASE) + boff[0][1], b02 = (SBASE) + boff[0][2], b03 = (SBASE) + boff[0][3]; \
        const unsigned b10 = (SBASE) + boff[1][0], b11 = (SBASE) + boff[1][1], b12 = (SBASE) + boff[1][2], b13 = (SBASE) + boff[1][3]; \
        const bf16x8 f0 = CAT(C.l[0], C.h[0]), f1 = CAT(C.l[1], C.h[1]), f2 = CAT(C.l[2], C.h[2]), f3 = CAT(C.l[3], C.h[3]); \
        SB; M1(0, 0, f0, C.a[0]); SB; if (NX) { RTR(D.l[0], b00, BO); RTR(D.h[0], b10, BO); } \
        SB; M1(0, 1, f1, C.a[0]); SB; if (NX) { R128(D.a[0], aa, 0); } \
        SB; M1(0, 2, f2, C.a[0]); SB; if (NX) { RTR(D.l[1], b01, BO); RTR(D.h[1], b11, BO); } \
        SB; M1(0, 3, f3, C.a[0]); SB; if (NX) { R128(D.a[1], aa, 2048); } \
        SB; M1(1, 0, f0, C.a[1]); SB; if (NX) { RTR(D.l[2], b02, BO); RTR(D.h[2], b12, BO); } \
        SB; M1(1, 1, f1, C.a[1]); SB; if (NX) { R128(D.a[2], aa, 4096); } \
        SB; M1(1, 2, f2, C.a[1]); SB; if (NX) { RTR(D.l[3], b03, BO); RTR(D.h[3], b13, BO); } \
        SB; M1(1, 3, f3, C.a[1]); SB; if (NX) { R128(D.a[3], aa, 6144); } \
        SB; M1(2, 0, f0, C.a[2]); M1(2, 1, f1, C.a[2]); M1(2, 2, f2, C.a[2]); M1(2, 3, f3, C.a[2]); \
        M1(3, 0, f0, C.a[3]); M1(3, 1, f1, C.a[3]); M1(3, 2, f2, C.a[3]); M1(3, 3, f3, C.a[3]); SB; } while (0)
    __builtin_amdgcn_s_barrier();
    asm volatile("" ::: "memory");
    {
        const unsigned sb = smem_base;
#pragma unroll
        for (int nb = 0; nb < 4; ++nb) { RTR(X.l[nb], sb + boff[0][nb], 0); RTR(X.h[nb], sb + boff[1][nb], 0); }
        const unsigned aa = sb + aoff0;
        R128(X.a[0], aa, 0); R128(X.a[1], aa, 2048); R128(X.a[2], aa, 4096); R128(X.a[3], aa, 6144);
    }
    WL(X);
    int slot = 0;
    while (c < NST) {
        __builtin_amdgcn_s_barrier();
        asm volatile("" ::: "memory");
        int relv = NST;
        if (rel < NST) asm volatile("ds_read_b32 %0, %1\n\ts_waitcnt lgkmcnt(0)" : "=v"(relv) : "v"(relw + 4 * (it & 1)) : "memory");
        MACHINE_STEP(act, can);
        if (act) {
            const unsigned sb = smem_base + slot * SLOT_BYTES;
            const int nslot = slot == NSLOT - 1 ? 0 : slot + 1;
            const unsigned sn = smem_base + nslot * SLOT_BYTES;
            HALF(X, Y, sb, aoff1, 8192, true);
            WL(Y);
            HALF(Y, X, sn, aoff0, 0, true);
            WL(X);
            slot = nslot;
        }
        MACHINE_NEXT(act, can);
    }
#undef HALF
#undef M1
#undef WL
#undef CAT
#undef RTR
#undef R128
#undef SB
    if (wave == 0) STAMP(10);

    f32x4 keep[2][4];
    {
        char* xb = myws + WS_X + ((size_t)g * 4 + wave) * 8192 + lane * 16;
#pragma unroll
        for (int a = 0; a < 2; ++a)
#pragma unroll
            for (int b = 0; b < 4; ++b) {
                store16_agent(xb + (a * 4 + b) * 1024, h ? acc[a][b] : acc[2 + a][b]);
                keep[a][b] = h ? acc[2 + a][b] : acc[a][b];
            }
        asm volatile("s_waitcnt vmcnt(0)" ::: "memory");
    }
    __syncthreads();
    if (tid == 0)
        __hip_atomic_store((unsigned*)(myws + WS_XFLAG + g * 64), call, __ATOMIC_RELAXED, __HIP_MEMORY_SCOPE_AGENT);
    float* red = (float*)(smem + 16384);
    {
        const int gp = g ^ 8;
        while (ld_agent32(myws + WS_XFLAG + gp * 64) != call) {
            if (give_up(tmo, call, t0)) break;
            __builtin_amdgcn_s_sleep(1);
        }
        const char* xb = myws + WS_X + ((size_t)gp * 4 + wave) * 8192 + lane * 16;
        f32x4 r0, r1, r2, r3, r4, r5, r6, r7;
        asm volatile("global_load_dwordx4 %0, %8, off sc1\n\t"
                     "global_load_dwordx4 %1, %8, off offset:1024 sc1\n\t"
                     "global_load_dwordx4 %2, %8, off offset:2048 sc1\n\t"
                     "global_load_dwordx4 %3, %8, off offset:3072 sc1\n\t"
                     "global_load_dwordx4 %4, %9, off sc1\n\t"
                     "global_load_dwordx4 %5, %9, off offset:1024 sc1\n\t"
                     "global_load_dwordx4 %6, %9, off offset:2048 sc1\n\t"
                     "global_load_dwordx4 %7, %9, off offset:3072 sc1\n\t"
                     "s_waitcnt vmcnt(0)"
                     : "=&v"(r0), "=&v"(r1), "=&v"(r2), "=&v"(r3), "=&v"(r4), "=&v"(r5), "=&v"(r6), "=&v"(r7)
                     : "v"(xb), "v"(xb + 4096) : "memory");
        keep[0][0] += r0; keep[0][1] += r1; keep[0][2] += r2; keep[0][3] += r3;
        keep[1][0] += r4; keep[1][1] += r5; keep[1][2] += r6; keep[1][3] += r7;
        float m = 0.f;
#pragma unroll
        for (int a = 0; a < 2; ++a)
#pragma unroll
            for (int b = 0; b < 4; ++b)
#pragma unroll
                for (int r = 0; r < 4; ++r) m = fmaxf(m, fabsf(keep[a][b][r]));
#pragma unroll
        for (int o = 32; o > 0; o >>= 1) m = fmaxf(m, __shfl_xor(m, o));
        if (lane == 0) red[wave] = m;
    }
    __syncthreads();
    if (wave == 0) STAMP(11);

    if (wave == 0) {
        const float wgm = fmaxf(fmaxf(red[0], red[1]), fmaxf(red[2], red[3]));
        const unsigned long long tag = (unsigned long long)call << 32;
        unsigned long long* loc = (unsigned long long*)(myws + WS_LOCAL);
        if (lane == 0)
            __hip_atomic_store(loc + g, tag | __float_as_uint(wgm), __ATOMIC_RELAXED, __HIP_MEMORY_SCOPE_AGENT);
        if (idx == 0) {
            unsigned long long v;
            for (;;) {
                v = ld_agent64(loc + grp + 8 * (lane & 31));
                if (__all((unsigned)(v >> 32) == call)) break;
                if (give_up(tmo, call, t0)) break;
                __builtin_amdgcn_s_sleep(1);
            }
            unsigned gm = (unsigned)v;
#pragma unroll
            for (int o = 32; o > 0; o >>= 1) { unsigned x = __shfl_xor(gm, o); gm = x > gm ? x : gm; }
            __hip_atomic_store((unsigned long long*)(p.ws[lane >> 3] + WS_GRAN) + (lane & 7) * 64 + rank * 8 + grp, tag | gm,
                               __ATOMIC_RELAXED, __HIP_MEMORY_SCOPE_SYSTEM);
        }
        unsigned long long v;
        const unsigned long long* gran = (const unsigned long long*)(myws + WS_GRAN) + grp * 64;
        for (;;) {
            v = ld_sys64(gran + lane);
            if (__all((unsigned)(v >> 32) == call)) break;
            if (give_up(tmo, call, t0)) break;
            __builtin_amdgcn_s_sleep(1);
        }
        unsigned gm = (unsigned)v;
#pragma unroll
        for (int o = 32; o > 0; o >>= 1) { unsigned x = __shfl_xor(gm, o); gm = x > gm ? x : gm; }
        if (lane == 0) {
            red[4] = __uint_as_float(gm);
            ((unsigned*)red)[5] = (ld_agent32(tmo) == call) ? 1u : 0u;
            if (g == 0) __hip_atomic_store((unsigned*)(myws + WS_EPOCH), call, __ATOMIC_RELAXED, __HIP_MEMORY_SCOPE_AGENT);
        }
    }
    __syncthreads();
    if (wave == 0) STAMP(12);
    const float amax = red[4];
    const bool bad = ((unsigned*)red)[5] != 0u;
    const float inv = amax > 0.f ? 127.f / amax : 0.f;

    constexpr int OSTR = BN + 16;
    char* ot = smem;
#pragma unroll
    for (int mb = 0; mb < 2; ++mb)
#pragma unroll
        for (int nb = 0; nb < 4; ++nb) {
            unsigned pk = 0;
#pragma unroll
            for (int r = 0; r < 4; ++r) {
                float q = rintf(keep[mb][nb][r] * inv);
                q = fminf(fmaxf(q, -127.f), 127.f);
                int qi = bad ? -128 : (int)q;
                pk |= ((unsigned)qi & 0xffu) << (8 * r);
            }
            const int row = wm * 32 + mb * 16 + fr, col = wn * 64 + nb * 16 + fq * 4;
            *(unsigned*)(ot + row * OSTR + col) = pk;
        }
    __syncthreads();
#pragma unroll
    for (int i = 0; i < 2; ++i) {
        const int row = (tid >> 3) + 32 * i, c16 = tid & 7;
        const u32x4 v = *(const u32x4*)(ot + row * OSTR + c16 * 16);
        *(u32x4*)(p.out + (size_t)(mt * BM + i * 64 + h * 32 + (tid >> 3)) * NN + nt * BN + c16 * 16) = v;
    }
    if (wave == 0) STAMP(13);
}

}

extern "C" void kernel_launch(int rank, int nranks, ncclComm_t comm,
                              hipStream_t stream,
                              void* const* d_in, const int* in_sizes, int n_in,
                              void* d_out, int out_size,
                              const int* in_dtypes, int out_dtype,
                              void* const* d_out_all,
                              void* const* d_ws_all, size_t ws_bytes) {
    if (nranks != NR || n_in < 2 || ws_bytes < WS_END || (unsigned)rank >= (unsigned)NR) {
        fprintf(stderr, "unsupported configuration\n");
        return;
    }
    Params p;
    p.x = (const char*)d_in[0];
    p.w = (const char*)d_in[1];
    p.out = (signed char*)d_out;
    for (int i = 0; i < NR; ++i) p.ws[i] = (char*)d_ws_all[i];
    p.rank = rank;
    hipLaunchKernelGGL(a2a_gemm_q8, dim3(N_WG), dim3(NT), 0, stream, p);
    HIP_CHECK(hipGetLastError());
}
```

```cpp
#include <hip/hip_runtime.h>
#include <rccl/rccl.h>
#include <cstdio>
#include <cstdint>

#define HIP_CHECK(expr) do { hipError_t _e = (expr); if (_e != hipSuccess) { \
    fprintf(stderr, "HIP error %s at %s:%d\n", hipGetErrorString(_e), __FILE__, __LINE__); \
    return; } } while (0)

namespace {

typedef short bf16x8 __attribute__((ext_vector_type(8)));
typedef short s16x4 __attribute__((ext_vector_type(4)));
typedef float f32x4 __attribute__((ext_vector_type(4)));
typedef unsigned u32x4 __attribute__((ext_vector_type(4)));

constexpr int NR = 8;
constexpr int MLOC = 512;
constexpr int KB = 512;
constexpr int NN = 4096;
constexpr int CH = 64;
constexpr int NCH = KB / CH;
constexpr int BM = 128, BN = 128;
constexpr int NST = NR * NCH / 2;
constexpr int NOWN = NCH / 2;
constexpr int NSLOT = 4;
constexpr int A_BYTES = BM * CH * 2;
constexpr int B_BYTES = CH * BN * 2;
constexpr int SLOT_BYTES = A_BYTES + B_BYTES;
constexpr int N_WG = 256;
constexpr int NRG = MLOC / 16;
constexpr int NT = 640;
#define POLL_LOW 3
#define PUSH_WIN 1

constexpr size_t WS_A = 0;
constexpr size_t WS_X = (size_t)NR * KB * MLOC * 2;
constexpr size_t WS_FLAG = WS_X + (size_t)N_WG * 32768;
constexpr size_t WS_XFLAG = WS_FLAG + NR * NCH * 4 * 128;
constexpr size_t WS_GRAN = WS_XFLAG + N_WG * 64;
constexpr size_t WS_LOCAL = WS_GRAN + 8 * 512;
constexpr size_t WS_EPOCH = WS_LOCAL + 2048;
constexpr size_t WS_TMO = WS_EPOCH + 128;
constexpr size_t WS_END = WS_TMO + 128;

constexpr unsigned long long DEADLINE_TICKS = 50000000ull;

struct Params {
    const char* x;
    const char* w;
    signed char* out;
    char* ws[NR];
    int rank;
};

__device__ __forceinline__ unsigned ld_agent32(const void* p) {
    return __hip_atomic_load((const unsigned*)p, __ATOMIC_RELAXED, __HIP_MEMORY_SCOPE_AGENT);
}
__device__ __forceinline__ unsigned long long ld_sys64(const void* p) {
    return __hip_atomic_load((const unsigned long long*)p, __ATOMIC_RELAXED, __HIP_MEMORY_SCOPE_SYSTEM);
}
__device__ __forceinline__ unsigned long long ld_agent64(const void* p) {
    return __hip_atomic_load((const unsigned long long*)p, __ATOMIC_RELAXED, __HIP_MEMORY_SCOPE_AGENT);
}

__device__ __forceinline__ bool give_up(unsigned* tmo, unsigned call, unsigned long long t0) {
    if (ld_agent32(tmo) == call || __builtin_amdgcn_s_memrealtime() - t0 > DEADLINE_TICKS) {
        __hip_atomic_store(tmo, call, __ATOMIC_RELAXED, __HIP_MEMORY_SCOPE_AGENT);
        return true;
    }
    return false;
}

__device__ __forceinline__ void store16_sys(void* p, u32x4 v) {
    asm volatile("global_store_dwordx4 %0, %1, off sc0 sc1\n\ts_nop 1" :: "v"(p), "v"(v) : "memory");
}
__device__ __forceinline__ void store16_agent(void* p, f32x4 v) {
    asm volatile("global_store_dwordx4 %0, %1, off sc1\n\ts_nop 1" :: "v"(p), "v"(v) : "memory");
}
__device__ __forceinline__ void dma16(unsigned voff, const char* sbase, unsigned lds) {
    asm volatile("s_mov_b32 m0, %2\n\ts_nop 0\n\tglobal_load_lds_dwordx4 %0, %1" :: "v"(voff), "s"(sbase), "s"(lds) : "memory");
}
__device__ __forceinline__ unsigned vm_outstanding() {
    unsigned s;
    asm volatile("s_getreg_b32 %0, hwreg(HW_REG_IB_STS)" : "=s"(s) :: "memory");
    return (s & 15u) | ((s >> 18) & 48u);
}

#define STAMP(i) do { } while (0)

__device__ __forceinline__ int swzB(int row) { return ((row & 3) << 2) | ((row >> 2) & 3); }

struct Frag { bf16x8 a[4]; s16x4 l[4], h[4]; };

__global__ void __launch_bounds__(NT, 3) a2a_gemm_q8(Params p) {
    __shared__ __attribute__((aligned(16))) char smem[NSLOT * SLOT_BYTES + 64 + 128 + 1024];
    const int tid = threadIdx.x, lane = tid & 63;
    const int wave = __builtin_amdgcn_readfirstlane(tid >> 6);
    const int rank = p.rank;
    char* myws = p.ws[rank];
    unsigned* tmo = (unsigned*)(myws + WS_TMO);
    const unsigned long long t0 = __builtin_amdgcn_s_memrealtime();

    const int g = blockIdx.x;
    const int grp = g & 7, idx = g >> 3;
    const int h = idx & 1, nt = grp * 4 + ((idx >> 1) & 3), mt = idx >> 3;
    const unsigned smem_base = (unsigned)(size_t)smem;
    const unsigned relw = smem_base + NSLOT * SLOT_BYTES;
    int c = 0, iss = NSLOT - 1, rel = NOWN, it = 0;
#define MACHINE_STEP(ACTIVE, CAN) \
        const bool ACTIVE = c + 1 < iss || c + 1 >= NST; \
        const int CAN = max(0, min(min(rel, c + NSLOT), NST) - iss);
#define MACHINE_NEXT(ACTIVE, CAN) do { rel = __builtin_amdgcn_readfirstlane(relv); iss += CAN; c += ACTIVE ? 1 : 0; ++it; } while (0)

    if (wave == 8) {
        const int j = g / NRG, rg = g % NRG;
        const int dst = (rank + 1 + j) & (NR - 1);
        const bool active = g < 7 * NRG && (unsigned)dst < (unsigned)NR;
        char* dws = p.ws[active ? dst : rank];
        const char* src = p.x + (size_t)((active ? dst : rank) * MLOC + rg * 16 + (lane >> 3)) * (KB * 2) + (lane & 7) * 16;
        char* dbase = dws + WS_A + ((size_t)rank * NCH * MLOC + rg * 16) * (CH * 2) + lane * 16;
        unsigned* flag = (unsigned*)(dws + WS_FLAG) + (rg >> 3) * 512 + rank * 8 + (rg & 7);
        STAMP(0);
        u32x4 v[NCH][2];
#pragma unroll
        for (int kc = 0; kc < NCH; ++kc)
#pragma unroll
            for (int q = 0; q < 2; ++q)
                asm volatile("global_load_dwordx4 %0, %1, off" : "=v"(v[kc][q]) : "v"(src + (size_t)q * 8 * (KB * 2) + kc * (CH * 2)) : "memory");
        unsigned epoch;
        asm volatile("global_load_dword %0, %1, off sc1" : "=v"(epoch) : "v"(myws + WS_EPOCH) : "memory");
        int niss = active ? 0 : NCH, npub = active ? 0 : NCH;
        unsigned nops = 0;
        unsigned long long endq = 0;
        asm volatile("s_waitcnt vmcnt(15)" : "+v"(v[0][0]), "+v"(v[0][1]) :: "memory");
        if (active) {
            store16_sys(dbase, v[0][0]); store16_sys(dbase + 1024, v[0][1]);
            nops = 2; endq = 2ull; niss = 1;
        }
        asm volatile("s_waitcnt vmcnt(2)"
                     : "+v"(v[1][0]), "+v"(v[1][1]), "+v"(v[2][0]), "+v"(v[2][1]), "+v"(v[3][0]), "+v"(v[3][1]), "+v"(v[4][0]), "+v"(v[4][1]),
                       "+v"(v[5][0]), "+v"(v[5][1]), "+v"(v[6][0]), "+v"(v[6][1]), "+v"(v[7][0]), "+v"(v[7][1]), "+v"(epoch) :: "memory");
        const unsigned call = __builtin_amdgcn_readfirstlane(epoch) + 1u;
        auto progress = [&]() {
            while (npub < niss && vm_outstanding() <= nops - (unsigned)((endq >> (8 * npub)) & 0xffull)) {
                if (lane == 0) __hip_atomic_store(flag + ((npub + rank) & 1) * 256 + (npub >> 1) * 64, call, __ATOMIC_RELAXED, __HIP_MEMORY_SCOPE_SYSTEM);
                asm volatile("" ::: "memory");
                ++npub; ++nops;
            }
#pragma unroll
            for (int kc = 1; kc < NCH; ++kc)
                if (niss == kc && kc < npub + PUSH_WIN) {
                    store16_sys(dbase + (size_t)kc * MLOC * (CH * 2), v[kc][0]);
                    store16_sys(dbase + (size_t)kc * MLOC * (CH * 2) + 1024, v[kc][1]);
                    nops += 2; endq |= (unsigned long long)nops << (8 * kc); ++niss;
                }
        };
        progress();
        __builtin_amdgcn_s_barrier();
        while (c < NST && npub < NCH) {
            progress();
            __builtin_amdgcn_s_barrier();
            asm volatile("" ::: "memory");
            int relv = NST;
            if (rel < NST) asm volatile("ds_read_b32 %0, %1\n\ts_waitcnt lgkmcnt(0)" : "=v"(relv) : "v"(relw + 4 * (it & 1)) : "memory");
            MACHINE_STEP(act, can);
            MACHINE_NEXT(act, can);
        }
        while (npub < NCH) { progress(); __builtin_amdgcn_s_sleep(1); }
        return;
    }

    if (wave == 9) {
        const unsigned call = ld_agent32(myws + WS_EPOCH) + 1u;
        asm volatile("s_waitcnt vmcnt(0)" ::: "memory");
        const char* fbase = myws + WS_FLAG + (mt * 2 + h) * 1024;
        unsigned todo = ~(0x01010101u << rank);
        int prel = NOWN;
        bool pending = false;
        const unsigned ordw = relw + 64, pbuf = relw + 64 + 128;
        const unsigned pvo = lane * 16;
#define POLL_ISSUE() do { \
            asm volatile("s_mov_b32 m0, %2\n\ts_nop 0\n\tglobal_load_lds_dwordx4 %0, %1 sc0 sc1" \
                         :: "v"(pvo), "s"(fbase), "s"(pbuf) : "memory"); \
            pending = true; } while (0)
        POLL_ISSUE();
        __builtin_amdgcn_s_barrier();
        while (c < NST && rel < NST) {
            if (pending && vm_outstanding() == 0) {
                u32x4 f, f2;
                asm volatile("ds_read_b128 %0, %2\n\tds_read_b128 %1, %2 offset:16\n\ts_waitcnt lgkmcnt(0)"
                             : "=&v"(f), "=&v"(f2) : "v"(pbuf + (lane & 31) * 32) : "memory");
                const unsigned lo = min(min(min(f[0], f[1]), min(f[2], f[3])), min(min(f2[0], f2[1]), min(f2[2], f2[3])));
                unsigned fresh = (unsigned)__ballot(lo >= call) & todo;
                pending = false;
                todo &= ~fresh;
                while (fresh) {
                    const int b = __builtin_ctz(fresh);
                    asm volatile("ds_write_b32 %0, %1" :: "v"(ordw + 4 * prel), "v"(b) : "memory");
                    fresh &= fresh - 1u; ++prel;
                }
            }
            if (prel < NST && !pending && prel - iss < POLL_LOW) {
                if ((it & 63) == 63 && give_up(tmo, call, t0)) {
                    for (; prel < NST; ++prel) asm volatile("ds_write_b32 %0, %1" :: "v"(ordw + 4 * prel), "v"(rank) : "memory");
                } else POLL_ISSUE();
            }
            asm volatile("ds_write_b32 %0, %1\n\ts_waitcnt lgkmcnt(0)" :: "v"(relw + 4 * (it & 1)), "v"(prel) : "memory");
            __builtin_amdgcn_s_barrier();
            asm volatile("" ::: "memory");
            const int relv = prel;
            MACHINE_STEP(act, can);
            MACHINE_NEXT(act, can);
        }
#undef POLL_ISSUE
        asm volatile("s_waitcnt vmcnt(0)" ::: "memory");
        return;
    }

    if (wave >= 4) {
        const int lw = wave - 4;
        unsigned avo_own[4], avo_rem[4], bvo[4];
#pragma unroll
        for (int i = 0; i < 4; ++i) {
            const int row = lw * 32 + i * 8 + (lane >> 3);
            const int ch = ((lane & 7) ^ ((row >> 1) & 7)) << 4;
            avo_own[i] = row * (KB * 2) + ch;
            avo_rem[i] = row * (CH * 2) + ch;
            const int brow = lw * 16 + i * 4 + (lane >> 4);
            bvo[i] = brow * (NN * 2) + (((lane & 15) ^ swzB(brow)) << 4);
        }
        const char* a_own = p.x + (size_t)(rank * MLOC + mt * BM) * (KB * 2) + ((h + rank) & 1) * (CH * 2);
        const char* a_rem = myws + WS_A + (size_t)(mt * BM) * (CH * 2);
        const char* b0 = p.w + nt * (BN * 2);
        int islot = 0;
        auto issue = [&](int s) {
            const unsigned buf = smem_base + islot * SLOT_BYTES + lw * 4096;
            if (s < NOWN) {
                const char* ab = a_own + s * (2 * CH * 2);
                const char* bb = b0 + (size_t)(rank * KB + (2 * s + ((h + rank) & 1)) * CH) * (NN * 2);
#pragma unroll
                for (int i = 0; i < 4; ++i) { dma16(avo_own[i], ab, buf + i * 1024); dma16(bvo[i], bb, buf + A_BYTES + i * 1024); }
            } else {
                int o;
                asm volatile("ds_read_b32 %0, %1\n\ts_waitcnt lgkmcnt(0)" : "=v"(o) : "v"(relw + 64 + 4 * s) : "memory");
                o = __builtin_amdgcn_readfirstlane(o);
                const int ir = (o >> 3) & 3, src = o & (NR - 1);
                const int kc = 2 * ir + ((h + src) & 1);
                const char* ab = a_rem + (size_t)(src * NCH + kc) * MLOC * (CH * 2);
                const char* bb = b0 + (size_t)(src * KB + kc * CH) * (NN * 2);
#pragma unroll
                for (int i = 0; i < 4; ++i) { dma16(avo_rem[i], ab, buf + i * 1024); dma16(bvo[i], bb, buf + A_BYTES + i * 1024); }
            }
            islot = islot == NSLOT - 1 ? 0 : islot + 1;
        };
#pragma unroll
        for (int s = 0; s < NSLOT - 1; ++s) issue(s);
        asm volatile("s_waitcnt vmcnt(%0)" :: "n"(8 * (NSLOT - 2)) : "memory");
        __builtin_amdgcn_s_barrier();
        while (c < NST) {
            if (iss >= c + 4) asm volatile("s_waitcnt vmcnt(16)" ::: "memory");
            else if (iss == c + 3) asm volatile("s_waitcnt vmcnt(8)" ::: "memory");
            else asm volatile("s_waitcnt vmcnt(0)" ::: "memory");
            __builtin_amdgcn_s_barrier();
            asm volatile("" ::: "memory");
            MACHINE_STEP(act, can);
            for (int k = 0; k < can; ++k) issue(iss + k);
            int relv = NST;
            if (rel < NST) asm volatile("ds_read_b32 %0, %1\n\ts_waitcnt lgkmcnt(0)" : "=v"(relv) : "v"(relw + 4 * (it & 1)) : "memory");
            MACHINE_NEXT(act, can);
        }
        return;
    }

    STAMP(0);
    const unsigned call = ld_agent32(myws + WS_EPOCH) + 1u;
    const int wm = wave >> 1, wn = wave & 1;
    f32x4 acc[4][4];
#pragma unroll
    for (int a = 0; a < 4; ++a)
#pragma unroll
        for (int b = 0; b < 4; ++b) acc[a][b] = (f32x4){0.f, 0.f, 0.f, 0.f};
    const int fr = lane & 15, fq = lane >> 4;
    const int aoff0 = (wm * 64 + fr) * 128 + (((0 + fq) ^ ((fr >> 1) & 7)) << 4);
    const int aoff1 = (wm * 64 + fr) * 128 + (((4 + fq) ^ ((fr >> 1) & 7)) << 4);
    int boff[2][4];
#pragma unroll
    for (int t = 0; t < 2; ++t) {
        const int row = 8 * fq + 4 * t + ((lane & 15) >> 2);
#pragma unroll
        for (int nb = 0; nb < 4; ++nb)
            boff[t][nb] = A_BYTES + row * 256 + 8 * (lane & 1) + (((wn * 8 + 2 * nb + ((lane & 3) >> 1)) ^ swzB(row)) << 4);
    }
    Frag X, Y;
#define SB __builtin_amdgcn_sched_barrier(0)
#define R128(D, ADDR, OFF) asm volatile("ds_read_b128 %0, %1 offset:" #OFF : "=v"(D) : "v"(ADDR) : "memory")
#define RTR(D, ADDR, OFF) asm volatile("ds_read_b64_tr_b16 %0, %1 offset:" #OFF : "=v"(D) : "v"(ADDR) : "memory")
#define CAT(lo, hi) __builtin_shufflevector(lo, hi, 0, 1, 2, 3, 4, 5, 6, 7)
#define WL(F) asm volatile("s_waitcnt lgkmcnt(0)" : "+v"(F.a[0]), "+v"(F.a[1]), "+v"(F.a[2]), "+v"(F.a[3]), \
        "+v"(F.l[0]), "+v"(F.h[0]), "+v"(F.l[1]), "+v"(F.h[1]), "+v"(F.l[2]), "+v"(F.h[2]), "+v"(F.l[3]), "+v"(F.h[3]) :: "memory")
#define M1(I, J, F, A) acc[I][J] = __builtin_amdgcn_mfma_f32_16x16x32_bf16(F, A, acc[I][J], 0, 0, 0)
#define HALF(C, D, SBASE, AO, BO, NX) do { \
        const unsigned aa = (SBASE) + (AO), b00 = (SBASE) + boff[0][0], b01 = (SBASE) + boff[0][1], b02 = (SBASE) + boff[0][2], b03 = (SBASE) + boff[0][3]; \
        const unsigned b10 = (SBASE) + boff[1][0], b11 = (SBASE) + boff[1][1], b12 = (SBASE) + boff[1][2], b13 = (SBASE) + boff[1][3]; \
        const bf16x8 f0 = CAT(C.l[0], C.h[0]), f1 = CAT(C.l[1], C.h[1]), f2 = CAT(C.l[2], C.h[2]), f3 = CAT(C.l[3], C.h[3]); \
        SB; M1(0, 0, f0, C.a[0]); SB; if (NX) { RTR(D.l[0], b00, BO); RTR(D.h[0], b10, BO); } \
        SB; M1(0, 1, f1, C.a[0]); SB; if (NX) { R128(D.a[0], aa, 0); } \
        SB; M1(0, 2, f2, C.a[0]); SB; if (NX) { RTR(D.l[1], b01, BO); RTR(D.h[1], b11, BO); } \
        SB; M1(0, 3, f3, C.a[0]); SB; if (NX) { R128(D.a[1], aa, 2048); } \
        SB; M1(1, 0, f0, C.a[1]); SB; if (NX) { RTR(D.l[2], b02, BO); RTR(D.h[2], b12, BO); } \
        SB; M1(1, 1, f1, C.a[1]); SB; if (NX) { R128(D.a[2], aa, 4096); } \
        SB; M1(1, 2, f2, C.a[1]); SB; if (NX) { RTR(D.l[3], b03, BO); RTR(D.h[3], b13, BO); } \
        SB; M1(1, 3, f3, C.a[1]); SB; if (NX) { R128(D.a[3], aa, 6144); } \
        SB; M1(2, 0, f0, C.a[2]); M1(2, 1, f1, C.a[2]); M1(2, 2, f2, C.a[2]); M1(2, 3, f3, C.a[2]); \
        M1(3, 0, f0, C.a[3]); M1(3, 1, f1, C.a[3]); M1(3, 2, f2, C.a[3]); M1(3, 3, f3, C.a[3]); SB; } while (0)
    __builtin_amdgcn_s_barrier();
    asm volatile("" ::: "memory");
    {
        const unsigned sb = smem_base;
#pragma unroll
        for (int nb = 0; nb < 4; ++nb) { RTR(X.l[nb], sb + boff[0][nb], 0); RTR(X.h[nb], sb + boff[1][nb], 0); }
        const unsigned aa = sb + aoff0;
        R128(X.a[0], aa, 0); R128(X.a[1], aa, 2048); R128(X.a[2], aa, 4096); R128(X.a[3], aa, 6144);
    }
    WL(X);
    int slot = 0;
    while (c < NST) {
        __builtin_amdgcn_s_barrier();
        asm volatile("" ::: "memory");
        int relv = NST;
        if (rel < NST) asm volatile("ds_read_b32 %0, %1\n\ts_waitcnt lgkmcnt(0)" : "=v"(relv) : "v"(relw + 4 * (it & 1)) : "memory");
        MACHINE_STEP(act, can);
        if (act) {
            const unsigned sb = smem_base + slot * SLOT_BYTES;
            const int nslot = slot == NSLOT - 1 ? 0 : slot + 1;
            const unsigned sn = smem_base + nslot * SLOT_BYTES;
            HALF(X, Y, sb, aoff1, 8192, true);
            WL(Y);
            HALF(Y, X, sn, aoff0, 0, true);
            WL(X);
            slot = nslot;
        }
        MACHINE_NEXT(act, can);
    }
#undef HALF
#undef M1
#undef WL
#undef CAT
#undef RTR
#undef R128
#undef SB
    if (wave == 0) STAMP(10);

    f32x4 keep[2][4];
    {
        char* xb = myws + WS_X + ((size_t)g * 4 + wave) * 8192 + lane * 16;
#pragma unroll
        for (int a = 0; a < 2; ++a)
#pragma unroll
            for (int b = 0; b < 4; ++b) {
                store16_agent(xb + (a * 4 + b) * 1024, h ? acc[a][b] : acc[2 + a][b]);
                keep[a][b] = h ? acc[2 + a][b] : acc[a][b];
            }
        asm volatile("s_waitcnt vmcnt(0)" ::: "memory");
    }
    __syncthreads();
    if (tid == 0)
        __hip_atomic_store((unsigned*)(myws + WS_XFLAG + g * 64), call, __ATOMIC_RELAXED, __HIP_MEMORY_SCOPE_AGENT);
    float* red = (float*)(smem + 16384);
    {
        const int gp = g ^ 8;
        for (unsigned spin = 1; ld_agent32(myws + WS_XFLAG + gp * 64) != call; ++spin)
            if ((spin & 31u) == 0u && give_up(tmo, call, t0)) break;
        const char* xb = myws + WS_X + ((size_t)gp * 4 + wave) * 8192 + lane * 16;
        f32x4 r0, r1, r2, r3, r4, r5, r6, r7;
        asm volatile("global_load_dwordx4 %0, %8, off sc1\n\t"
                     "global_load_dwordx4 %1, %8, off offset:1024 sc1\n\t"
                     "global_load_dwordx4 %2, %8, off offset:2048 sc1\n\t"
                     "global_load_dwordx4 %3, %8, off offset:3072 sc1\n\t"
                     "global_load_dwordx4 %4, %9, off sc1\n\t"
                     "global_load_dwordx4 %5, %9, off offset:1024 sc1\n\t"
                     "global_load_dwordx4 %6, %9, off offset:2048 sc1\n\t"
                     "global_load_dwordx4 %7, %9, off offset:3072 sc1\n\t"
                     "s_waitcnt vmcnt(0)"
                     : "=&v"(r0), "=&v"(r1), "=&v"(r2), "=&v"(r3), "=&v"(r4), "=&v"(r5), "=&v"(r6), "=&v"(r7)
                     : "v"(xb), "v"(xb + 4096) : "memory");
        keep[0][0] += r0; keep[0][1] += r1; keep[0][2] += r2; keep[0][3] += r3;
        keep[1][0] += r4; keep[1][1] += r5; keep[1][2] += r6; keep[1][3] += r7;
        float m = 0.f;
#pragma unroll
        for (int a = 0; a < 2; ++a)
#pragma unroll
            for (int b = 0; b < 4; ++b)
#pragma unroll
                for (int r = 0; r < 4; ++r) m = fmaxf(m, fabsf(keep[a][b][r]));
#pragma unroll
        for (int o = 32; o > 0; o >>= 1) m = fmaxf(m, __shfl_xor(m, o));
        if (lane == 0) red[wave] = m;
    }
    __syncthreads();
    if (wave == 0) STAMP(11);

    if (wave == 0) {
        const float wgm = fmaxf(fmaxf(red[0], red[1]), fmaxf(red[2], red[3]));
        const unsigned long long tag = (unsigned long long)call << 32;
        unsigned long long* loc = (unsigned long long*)(myws + WS_LOCAL);
        if (lane == 0)
            __hip_atomic_store(loc + g, tag | __float_as_uint(wgm), __ATOMIC_RELAXED, __HIP_MEMORY_SCOPE_AGENT);
        if (idx == 0) {
            unsigned long long v;
            for (unsigned spin = 1;; ++spin) {
                v = ld_agent64(loc + grp + 8 * (lane & 31));
                if (__all((unsigned)(v >> 32) == call)) break;
                if ((spin & 31u) == 0u && give_up(tmo, call, t0)) break;
            }
            unsigned gm = (unsigned)v;
#pragma unroll
            for (int o = 32; o > 0; o >>= 1) { unsigned x = __shfl_xor(gm, o); gm = x > gm ? x : gm; }
            __hip_atomic_store((unsigned long long*)(p.ws[lane >> 3] + WS_GRAN) + (lane & 7) * 64 + rank * 8 + grp, tag | gm,
                               __ATOMIC_RELAXED, __HIP_MEMORY_SCOPE_SYSTEM);
        }
        unsigned long long v;
        const unsigned long long* gran = (const unsigned long long*)(myws + WS_GRAN) + grp * 64;
        for (unsigned spin = 1;; ++spin) {
            v = ld_sys64(gran + lane);
            if (__all((unsigned)(v >> 32) == call)) break;
            if ((spin & 31u) == 0u && give_up(tmo, call, t0)) break;
            __builtin_amdgcn_s_sleep(1);
        }
        unsigned gm = (unsigned)v;
#pragma unroll
        for (int o = 32; o > 0; o >>= 1) { unsigned x = __shfl_xor(gm, o); gm = x > gm ? x : gm; }
        if (lane == 0) {
            red[4] = __uint_as_float(gm);
            ((unsigned*)red)[5] = (ld_agent32(tmo) == call) ? 1u : 0u;
            if (g == 0) __hip_atomic_store((unsigned*)(myws + WS_EPOCH), call, __ATOMIC_RELAXED, __HIP_MEMORY_SCOPE_AGENT);
        }
    }
    __syncthreads();
    if (wave == 0) STAMP(12);
    const float amax = red[4];
    const bool bad = ((unsigned*)red)[5] != 0u;
    const float inv = amax > 0.f ? 127.f / amax : 0.f;

    constexpr int OSTR = BN + 16;
    char* ot = smem;
#pragma unroll
    for (int mb = 0; mb < 2; ++mb)
#pragma unroll
        for (int nb = 0; nb < 4; ++nb) {
            unsigned pk = 0;
#pragma unroll
            for (int r = 0; r < 4; ++r) {
                float q = rintf(keep[mb][nb][r] * inv);
                q = fminf(fmaxf(q, -127.f), 127.f);
                int qi = bad ? -128 : (int)q;
                pk |= ((unsigned)qi & 0xffu) << (8 * r);
            }
            const int row = wm * 32 + mb * 16 + fr, col = wn * 64 + nb * 16 + fq * 4;
            *(unsigned*)(ot + row * OSTR + col) = pk;
        }
    __syncthreads();
#pragma unroll
    for (int i = 0; i < 2; ++i) {
        const int row = (tid >> 3) + 32 * i, c16 = tid & 7;
        const u32x4 v = *(const u32x4*)(ot + row * OSTR + c16 * 16);
        *(u32x4*)(p.out + (size_t)(mt * BM + i * 64 + h * 32 + (tid >> 3)) * NN + nt * BN + c16 * 16) = v;
    }
    if (wave == 0) STAMP(13);
}

}

extern "C" void kernel_launch(int rank, int nranks, ncclComm_t comm,
                              hipStream_t stream,
                              void* const* d_in, const int* in_sizes, int n_in,
                              void* d_out, int out_size,
                              const int* in_dtypes, int out_dtype,
                              void* const* d_out_all,
                              void* const* d_ws_all, size_t ws_bytes) {
    if (nranks != NR || n_in < 2 || ws_bytes < WS_END || (unsigned)rank >= (unsigned)NR) {
        fprintf(stderr, "unsupported configuration\n");
        return;
    }
    Params p;
    p.x = (const char*)d_in[0];
    p.w = (const char*)d_in[1];
    p.out = (signed char*)d_out;
    for (int i = 0; i < NR; ++i) p.ws[i] = (char*)d_ws_all[i];
    p.rank = rank;
    hipLaunchKernelGGL(a2a_gemm_q8, dim3(N_WG), dim3(NT), 0, stream, p);
    HIP_CHECK(hipGetLastError());
}
```
